```python
import numpy as np

kernel_name = "gpu_gemm_rs_m2048_k2048_n2048_bf16_silu"


ABSMAX_RTOL = 0.008

COMPUTE_DTYPE = "bfloat16"
EPILOGUE = "silu"
IN_QUANT = None
OUT_QUANT = None
IN_DTYPES = {"x": "bfloat16", "w_mat": "bfloat16"}
WORLD_SIZE = 8

def _to_bf16(x):
    u = np.ascontiguousarray(x, dtype=np.float32).view(np.uint32)
    return ((u + 0x7FFF + ((u >> 16) & 1)) & 0xFFFF0000).view(np.float32)

def setup_inputs():

    x = _to_bf16(np.random.randn(2048, 2048).astype(np.float32))
    w_mat = _to_bf16((np.random.randn(2048, 2048) / np.sqrt(2048)).astype(np.float32))
    return {"x": x, "w_mat": w_mat}

def _epilogue(y):
    return (y / (1.0 + np.exp(-np.clip(y, -60.0, 60.0)))).astype(np.float32)

def reference(x, w_mat):
    y = (x @ w_mat).astype(np.float32)
    return _epilogue(y)
```

```cpp
#include <hip/hip_runtime.h>
#include <rccl/rccl.h>
#include <cstdio>

typedef unsigned u32x4 __attribute__((ext_vector_type(4)));
typedef unsigned u32x2 __attribute__((ext_vector_type(2)));
typedef short s16x4 __attribute__((ext_vector_type(4)));
typedef __bf16 bf16x8 __attribute__((ext_vector_type(8)));
typedef __bf16 bf16x2 __attribute__((ext_vector_type(2)));
typedef float f32x2 __attribute__((ext_vector_type(2)));
typedef float f32x4 __attribute__((ext_vector_type(4)));
typedef unsigned long long u64;
typedef __amdgpu_buffer_rsrc_t rsrc_t;
#define LDSP __attribute__((address_space(3)))

constexpr unsigned KIB = 1024u, MIB = 1024u * 1024u;
constexpr unsigned SLOT_BYTES = 512 * KIB;
constexpr unsigned IN_BYTES = 6 * SLOT_BYTES;
constexpr unsigned OUT_BYTES = 2 * MIB;
constexpr unsigned ROT_BYTES = IN_BYTES + OUT_BYTES;
constexpr unsigned CTR_OFF = 3 * ROT_BYTES;
constexpr unsigned WS_NEED = CTR_OFF + 64 * KIB;
constexpr unsigned X_BYTES = 2048u * 256u * 2u, W_BYTES = 256u * 2048u * 2u, Y_BYTES = 256u * 2048u * 2u;
constexpr u64 DEADLINE = 50000000ull;
constexpr int SYS = 17;
constexpr int LDS_BYTES = 128 * 1024;

struct Args {
  char* ws[8];
  const char* x; const char* w; char* out;
  int rank;
  int tx0, dslot, front, to0, zdelay, odelay, pad0, pad1;
};

__device__ __forceinline__ u64 now() { return __builtin_amdgcn_s_memrealtime(); }
__device__ __forceinline__ rsrc_t mk(const void* p, unsigned bytes) { return __builtin_amdgcn_make_buffer_rsrc((void*)p, 0, bytes, 0x00020000); }
__device__ __forceinline__ unsigned umin2(unsigned a, unsigned b) { return a < b ? a : b; }
__device__ __forceinline__ unsigned min4(u32x4 v) { return umin2(umin2(v.x, v.y), umin2(v.z, v.w)); }
__device__ __forceinline__ bool all_nz(unsigned m) { return __builtin_amdgcn_ballot_w64(m == 0u) == 0ull; }
__device__ __forceinline__ u32x4 nz(u32x4 v) {
  v.x = v.x ? v.x : 0x8000u; v.y = v.y ? v.y : 0x8000u; v.z = v.z ? v.z : 0x8000u; v.w = v.w ? v.w : 0x8000u; return v;
}
__device__ __forceinline__ unsigned pack2(float a, float b) { f32x2 f = {a, b}; return __builtin_bit_cast(unsigned, __builtin_convertvector(f, bf16x2)); }
__device__ __forceinline__ float lo16(unsigned u) { return __builtin_bit_cast(float, u << 16); }
__device__ __forceinline__ float hi16(unsigned u) { return __builtin_bit_cast(float, u & 0xffff0000u); }
__device__ __forceinline__ float silu(float y) {
  float c = __builtin_amdgcn_fmed3f(y, -60.f, 60.f);
  float e = __builtin_amdgcn_exp2f(-1.4426950408889634f * c);
  return y * __builtin_amdgcn_rcpf(1.f + e);
}
#define CLOBBER() asm volatile("" ::: "memory")
__device__ __forceinline__ void st128(u32x4 v, rsrc_t r, unsigned off) { __builtin_amdgcn_raw_buffer_store_b128(v, r, off, 0, SYS); }
__device__ __forceinline__ void wait_until(u64 t0, unsigned ticks) {
  for (;;) {
    const unsigned el = (unsigned)(now() - t0);
    if (el >= ticks) break;
    if (ticks - el > 60u) __builtin_amdgcn_s_sleep(16); else __builtin_amdgcn_s_sleep(1);
  }
}
__device__ __forceinline__ unsigned wimg(unsigned row, unsigned c16) { return row * 128u + ((c16 ^ ((row & 2u) | ((row >> 1) & 4u))) << 4); }
__device__ __forceinline__ u32x4 tr_frag(const char* img, unsigned row0, unsigned ni, unsigned lane) {
  const unsigned g = lane >> 4, q = (lane & 15u) >> 2, p = lane & 3u;
  const unsigned r0 = row0 + 8u * g + q;
  s16x4 a = __builtin_amdgcn_ds_read_tr16_b64_v4i16((LDSP s16x4*)(img + wimg(r0, 2u * ni + (p >> 1)) + 8u * (p & 1u)));
  s16x4 b = __builtin_amdgcn_ds_read_tr16_b64_v4i16((LDSP s16x4*)(img + wimg(r0 + 4u, 2u * ni + (p >> 1)) + 8u * (p & 1u)));
  u32x2 ua = __builtin_bit_cast(u32x2, a), ub = __builtin_bit_cast(u32x2, b);
  u32x4 r = {ua.x, ua.y, ub.x, ub.y};
  return r;
}
__device__ __forceinline__ unsigned slot_of(unsigned src, unsigned dst) {
  unsigned ps = src >> 1, pd = dst >> 1;
  return (ps - (ps > pd ? 1u : 0u)) * 2u + (src & 1u);
}
#define STAMP(i) do {} while (0)

__global__ void __launch_bounds__(512, 2) gemm_rs(Args a) {
  extern __shared__ __attribute__((aligned(16))) char smem[];
  const u64 t0 = now();
  const unsigned b = blockIdx.x;
  const unsigned lane = threadIdx.x & 63u;
  const unsigned wave = __builtin_amdgcn_readfirstlane(threadIdx.x >> 6);
  const unsigned rank = (unsigned)a.rank & 7u;
  const unsigned rm = rank >> 2, rn = (rank >> 1) & 1u, rk = rank & 1u;
  const unsigned j = b & 7u, c = (b >> 3) & 3u, qrel = (b >> 5) & 7u;
  const unsigned q = rank ^ qrel, rt = j >> 2, tj = 4u * c + (j & 3u);
  char* p_dst = a.ws[b < 24u ? 2u + b % 6u : qrel];
  unsigned p_tx0 = a.tx0, p_ds = a.dslot;
  asm volatile("" : "+s"(p_dst), "+s"(p_tx0), "+s"(p_ds));
  const rsrc_t rs_my = mk(a.ws[0], WS_NEED);
  const rsrc_t rs_x = mk(a.x, X_BYTES), rs_w = mk(a.w, W_BYTES);
  const unsigned callv = __builtin_amdgcn_raw_buffer_load_b32(rs_my, CTR_OFF + b * 4u, 0, SYS);
  unsigned call;
  bool dead = false;

  u32x4 wl[2][4];
  unsigned dbase = 0, wslot = 0;
  const rsrc_t rs_d = mk(p_dst, WS_NEED);
  const bool xsnd = qrel >= 2u && c == 3u, wsnd = b < 24u;
  auto w_burst = [&]() {
    #pragma unroll
    for (int h = 0; h < 2; ++h) {
      wait_until(t0, wslot + (h ? (16u * p_ds) >> 8 : 0u));
      #pragma unroll
      for (int ni = 0; ni < 4; ++ni) st128(wl[h][ni], rs_d, dbase + h * 16384u + ni * 1024u);
    }
    STAMP(2);
  };
  if (xsnd) {
    const unsigned u = wave * 8u + j;
    u32x4 xs[4];
    #pragma unroll
    for (int ks = 0; ks < 4; ++ks)
      xs[ks] = __builtin_amdgcn_raw_buffer_load_b128(rs_x, ((lane & 15u) * 256u + ks * 32u + (lane >> 4) * 8u) * 2u,
                                                     (((q ^ wave) * 256u + (q >> 2) * 128u + j * 16u) * 256u + (q & 1u) * 128u) * 2u, 0);
    CLOBBER();
    call = __builtin_amdgcn_readfirstlane(callv) + 1u;
    dbase = (call % 3u) * ROT_BYTES + slot_of(rank, q) * SLOT_BYTES + u * 4096u + lane * 16u;
    wait_until(t0, p_tx0 + ((u * 4u * p_ds) >> 8));
    #pragma unroll
    for (int ks = 0; ks < 4; ++ks) st128(nz(xs[ks]), rs_d, dbase + ks * 1024u);
    STAMP(0);
    wait_until(t0, p_tx0 + p_ds);
  } else if (wsnd) {
    const unsigned d = rank ^ (2u + b % 6u), k4 = b / 6u;
    wait_until(t0, a.odelay);
    #pragma unroll
    for (int h = 0; h < 2; ++h)
      #pragma unroll
      for (int i = 0; i < 4; ++i)
        wl[h][i] = __builtin_amdgcn_raw_buffer_load_b128(rs_w, ((i * 8u + (lane >> 3)) * 2048u + (lane & 7u) * 8u + h * 64u) * 2u,
                                                         (((d & 1u) * 128u + k4 * 32u) * 2048u + ((d >> 1) & 1u) * 1024u + wave * 128u) * 2u, 0);
    call = __builtin_amdgcn_readfirstlane(callv) + 1u;
    dbase = (call % 3u) * ROT_BYTES + slot_of(rank, d) * SLOT_BYTES + 256u * KIB + (wave * 8u + k4) * 4096u + lane * 16u;
    char* img = smem + wave * 16384u;
    #pragma unroll
    for (int h = 0; h < 2; ++h)
      #pragma unroll
      for (int i = 0; i < 4; ++i) *(LDSP u32x4*)(img + h * 4096u + wimg(i * 8u + (lane >> 3), lane & 7u)) = wl[h][i];
    #pragma unroll
    for (int h = 0; h < 2; ++h)
      #pragma unroll
      for (int ni = 0; ni < 4; ++ni) wl[h][ni] = nz(tr_frag(img + h * 4096u, 0, ni, lane));
    wslot = p_tx0 + (((256u + (wave * 8u + k4) * 4u) * p_ds) >> 8);
  } else {
    call = __builtin_amdgcn_readfirstlane(callv) + 1u;
    if (b < 64u) {
      wait_until(t0, a.zdelay);
      const unsigned zrot = (call + 2u) % 3u;
      const u32x4 z = {0u, 0u, 0u, 0u};
      #pragma unroll
      for (unsigned i = 0; i < 16u; ++i)
        __builtin_amdgcn_raw_buffer_store_b128(z, rs_my, (i * 320u + (b - 24u) * 8u + wave) * 1024u + lane * 16u + zrot * ROT_BYTES, 0, 0);
    }
  }

  const unsigned rot = call % 3u;
  u32x4 xf[4][4], wf[4][4];
  if (wave >= 6u) {
    const unsigned h = wave - 6u;
    wait_until(t0, a.odelay);
    char* img = smem + wave * 16384u;
    #pragma unroll
    for (int i = 0; i < 16; ++i)
      wf[i >> 2][i & 3] = __builtin_amdgcn_raw_buffer_load_b128(rs_w, ((i * 8u + (lane >> 3)) * 2048u + (lane & 7u) * 8u) * 2u,
                                                                (h * 128u * 2048u + rn * 1024u + tj * 64u) * 2u, 0);
    #pragma unroll
    for (int mi = 0; mi < 4; ++mi)
      #pragma unroll
      for (int ks = 0; ks < 4; ++ks)
        xf[mi][ks] = __builtin_amdgcn_raw_buffer_load_b128(rs_x, ((mi * 16u + (lane & 15u)) * 256u + ks * 32u + (lane >> 4) * 8u) * 2u,
                                                           ((q * 256u + rm * 128u + rt * 64u) * 256u + h * 128u) * 2u, 0);
    #pragma unroll
    for (int i = 0; i < 16; ++i) *(LDSP u32x4*)(img + wimg(i * 8u + (lane >> 3), lane & 7u)) = wf[i >> 2][i & 3];
    #pragma unroll
    for (int ks = 0; ks < 4; ++ks)
      #pragma unroll
      for (int ni = 0; ni < 4; ++ni) wf[ks][ni] = tr_frag(img, ks * 32u, ni, lane);
    asm volatile("s_waitcnt lgkmcnt(0)" ::: "memory");
  } else {
    const unsigned sbase = rot * ROT_BYTES + wave * SLOT_BYTES;
    const unsigned xblk = (qrel * 2u + rt) * 16384u;
    const unsigned xoff = xblk + lane * 16u, woff = 256u * KIB + tj * 16384u + lane * 16u;
    auto poll = [&](unsigned off, u64 limit) -> bool {
      for (;;) {
        unsigned v = __builtin_amdgcn_raw_buffer_load_b32(rs_my, off + 16256u + (lane & 31u) * 4u, sbase, SYS);
        if (all_nz(v)) return true;
        if (now() - t0 > limit) return false;
        __builtin_amdgcn_s_sleep(4); CLOBBER();
      }
    };
    bool pend = wsnd;
    bool tryx = poll(xblk, pend ? (u64)(p_tx0 + p_ds - 150u) : DEADLINE);
    STAMP(3);
    bool xok = false, tryw = false;
    for (;;) {
      if (tryx) {
        #pragma unroll
        for (int i = 0; i < 16; ++i) xf[i >> 2][i & 3] = __builtin_amdgcn_raw_buffer_load_b128(rs_my, xoff + i * 1024u, sbase, SYS);
      }
      if (tryw) {
        #pragma unroll
        for (int i = 0; i < 16; ++i) wf[i >> 2][i & 3] = __builtin_amdgcn_raw_buffer_load_b128(rs_my, woff + i * 1024u, sbase, SYS);
      }
      if (tryx) {
        unsigned m = ~0u;
        #pragma unroll
        for (int i = 0; i < 16; ++i) m = umin2(m, min4(xf[i >> 2][i & 3]));
        xok = all_nz(m);
        if (xok) STAMP(4);
      }
      if (tryw) {
        unsigned m = ~0u;
        #pragma unroll
        for (int i = 0; i < 16; ++i) m = umin2(m, min4(wf[i >> 2][i & 3]));
        if (xok && all_nz(m)) break;
      }
      if (pend) { w_burst(); wait_until(t0, p_tx0 + 2u * p_ds); pend = false; }
      if (now() - t0 > DEADLINE) { dead = true; break; }
      tryx = !xok;
      if (xok && !tryw) { poll(256u * KIB + tj * 16384u, DEADLINE); tryw = true; STAMP(5); }
      else __builtin_amdgcn_s_sleep(8);
      CLOBBER();
    }
    STAMP(6);
  }
  f32x4 acc[4][4];
  {
    const float init = dead ? __builtin_nanf("") : 0.f;
    #pragma unroll
    for (int mi = 0; mi < 4; ++mi)
      #pragma unroll
      for (int ni = 0; ni < 4; ++ni) acc[mi][ni] = (f32x4){init, init, init, init};
  }
  #pragma unroll
  for (int ks = 0; ks < 4; ++ks)
    #pragma unroll
    for (int mi = 0; mi < 4; ++mi)
      #pragma unroll
      for (int ni = 0; ni < 4; ++ni)
        acc[mi][ni] = __builtin_amdgcn_mfma_f32_16x16x32_bf16(__builtin_bit_cast(bf16x8, wf[ks][ni]), __builtin_bit_cast(bf16x8, xf[mi][ks]), acc[mi][ni], 0, 0, 0);
  {
    char* slab = smem + wave * 16384u;
    #pragma unroll
    for (int mi = 0; mi < 4; ++mi)
      #pragma unroll
      for (int ni = 0; ni < 4; ++ni) {
        const unsigned nq = ni * 4u + (lane >> 4);
        *(LDSP f32x4*)(slab + (((mi * 16u + nq) * 16u + ((lane & 15u) ^ (nq & 14u))) << 4)) = acc[mi][ni];
      }
  }
  STAMP(7);
  if (wsnd && wave >= 6u) w_burst();
  asm volatile("s_waitcnt lgkmcnt(0)" ::: "memory");
  __builtin_amdgcn_s_barrier();
  CLOBBER();
  f32x4 s0 = {0.f, 0.f, 0.f, 0.f}, s1 = s0;
  {
    const unsigned m = wave * 8u + (lane >> 3), o = lane & 7u;
    const unsigned ad = (((m >> 4) * 16u + 2u * o) * 16u + ((m & 15u) ^ (2u * o))) << 4;
    #pragma unroll
    for (int s = 0; s < 8; ++s) {
      s0 += *(LDSP f32x4*)(smem + s * 16384u + ad);
      s1 += *(LDSP f32x4*)(smem + s * 16384u + ad + 256u);
    }
  }
  STAMP(8);
  const unsigned strip = wave * 1024u + lane * 16u;
  unsigned omh, onh, otj;
  bool two_parts;
  if (qrel != 0u) {
    u32x4 p = {pack2(s0.x, s0.y), pack2(s0.z, s0.w), pack2(s1.x, s1.y), pack2(s1.z, s1.w)};
    wait_until(t0, a.to0 + ((((tj * 2u + rt) * 8u + wave) * (unsigned)a.dslot) >> 8));
    st128(nz(p), rs_d, strip + rot * ROT_BYTES + IN_BYTES + rk * MIB + (((rm * 2u + rt) * 32u + rn * 16u + tj) << 13));
    STAMP(9);
    const unsigned quad = qrel >> 1;
    omh = rm ^ (quad >> 1); onh = rn ^ (quad & 1u);
    otj = 4u * ((c == 0u ? 2u : 0u) + (qrel & 1u)) + (j & 3u);
    two_parts = true;
    s0 = (f32x4){0.f, 0.f, 0.f, 0.f}; s1 = s0;
  } else {
    omh = rm; onh = rn; otj = tj; two_parts = false;
  }
  if (qrel == 0u || (qrel >= 2u && c < 2u)) {
    const unsigned tbase = rot * ROT_BYTES + IN_BYTES + (((omh * 2u + rt) * 32u + onh * 16u + otj) << 13);
    const unsigned pa = tbase + (rk ^ 1u) * MIB, pb = tbase + (two_parts ? rk : (rk ^ 1u)) * MIB;
    u32x4 u, v;
    dead = false;
    for (;;) {
      u = __builtin_amdgcn_raw_buffer_load_b128(rs_my, strip, pa, SYS);
      v = __builtin_amdgcn_raw_buffer_load_b128(rs_my, strip, pb, SYS);
      if (all_nz(umin2(min4(u), min4(v)))) break;
      if (now() - t0 > DEADLINE) { dead = true; break; }
      __builtin_amdgcn_s_sleep(2); CLOBBER();
    }
    STAMP(10);
    if (!two_parts) v = (u32x4){0u, 0u, 0u, 0u};
    float y0 = s0.x + lo16(u.x) + lo16(v.x), y1 = s0.y + hi16(u.x) + hi16(v.x), y2 = s0.z + lo16(u.y) + lo16(v.y), y3 = s0.w + hi16(u.y) + hi16(v.y);
    float y4 = s1.x + lo16(u.z) + lo16(v.z), y5 = s1.y + hi16(u.z) + hi16(v.z), y6 = s1.z + lo16(u.w) + lo16(v.w), y7 = s1.w + hi16(u.w) + hi16(v.w);
    u32x4 o = {pack2(silu(y0), silu(y1)), pack2(silu(y2), silu(y3)), pack2(silu(y4), silu(y5)), pack2(silu(y6), silu(y7))};
    if (dead) o = (u32x4){0x7fc07fc0u, 0x7fc07fc0u, 0x7fc07fc0u, 0x7fc07fc0u};
    const unsigned ooff = ((wave * 8u + (lane >> 3)) * 2048u + (lane & 7u) * 8u) * 2u + ((omh * 128u + rt * 64u) * 2048u + onh * 1024u + otj * 64u) * 2u;
    if (a.pad1 & 4) __builtin_amdgcn_raw_buffer_store_b128(o, mk(a.out, Y_BYTES), ooff, 0, 0);
    else st128(o, mk(a.out, Y_BYTES), ooff);
  }
  if (threadIdx.x == 0) __builtin_amdgcn_raw_buffer_store_b32(call, rs_my, CTR_OFF + b * 4u, 0, SYS);
}

static const int kDefaults[8] = {50, 435, 64, 920, 120, 120, 1, 4};

extern "C" void kernel_launch(int rank, int nranks, ncclComm_t comm, hipStream_t stream,
                              void* const* d_in, const int* in_sizes, int n_in,
                              void* d_out, int out_size, const int* in_dtypes, int out_dtype,
                              void* const* d_out_all, void* const* d_ws_all, size_t ws_bytes) {
  if (nranks != 8 || ws_bytes < WS_NEED || n_in < 2) { fprintf(stderr, "unexpected configuration\n"); return; }
  static bool init = false;
  const int lds = LDS_BYTES;
  if (!init) { (void)hipFuncSetAttribute((const void*)gemm_rs, hipFuncAttributeMaxDynamicSharedMemorySize, lds); init = true; }
  Args a;
  for (int p = 0; p < 8; ++p) a.ws[p] = (char*)d_ws_all[(rank ^ p) & 7];
  a.x = (const char*)d_in[0]; a.w = (const char*)d_in[1]; a.out = (char*)d_out; a.rank = rank;
  const int* cf = kDefaults;
  a.tx0 = cf[0]; a.dslot = cf[1]; a.front = cf[2]; a.to0 = cf[3]; a.zdelay = cf[4]; a.odelay = cf[5]; a.pad0 = cf[6]; a.pad1 = cf[7];
  gemm_rs<<<256, 512, lds, stream>>>(a);
}
```

```cpp
#include <hip/hip_runtime.h>
#include <rccl/rccl.h>
#include <cstdio>

#define TROT 1
#define XBIND 0
typedef unsigned u32x4 __attribute__((ext_vector_type(4)));
typedef unsigned u32x2 __attribute__((ext_vector_type(2)));
typedef short s16x4 __attribute__((ext_vector_type(4)));
typedef __bf16 bf16x8 __attribute__((ext_vector_type(8)));
typedef __bf16 bf16x2 __attribute__((ext_vector_type(2)));
typedef float f32x2 __attribute__((ext_vector_type(2)));
typedef float f32x4 __attribute__((ext_vector_type(4)));
typedef unsigned long long u64;
typedef __amdgpu_buffer_rsrc_t rsrc_t;
#define LDSP __attribute__((address_space(3)))

constexpr unsigned KIB = 1024u, MIB = 1024u * 1024u;
constexpr unsigned SLOT_BYTES = 512 * KIB;
constexpr unsigned IN_BYTES = 6 * SLOT_BYTES;
constexpr unsigned OUT_BYTES = 2 * MIB;
constexpr unsigned ROT_BYTES = IN_BYTES + OUT_BYTES;
constexpr unsigned CTR_OFF = 3 * ROT_BYTES;
constexpr unsigned WS_NEED = CTR_OFF + 64 * KIB;
constexpr unsigned X_BYTES = 2048u * 256u * 2u, W_BYTES = 256u * 2048u * 2u, Y_BYTES = 256u * 2048u * 2u;
constexpr u64 DEADLINE = 50000000ull;
constexpr int SYS = 17;
constexpr int LDS_TW = 128 * 1024;
constexpr int LDS_BYTES = LDS_TW + 64;
constexpr unsigned CODE_PF = 9728u;
constexpr unsigned XSLOTS = 224u;

struct Args {
  const char* x; char* my; const char* w; char* out;
  int rank;
  int tx0, dslot, front, to0, zdelay, odelay, pad0;
  char* ws[8];
  int pad1, dso, pgate;
};

__device__ __forceinline__ u64 now() { return __builtin_amdgcn_s_memrealtime(); }
__device__ __forceinline__ rsrc_t mk(const void* p, unsigned bytes) { return __builtin_amdgcn_make_buffer_rsrc((void*)p, 0, bytes, 0x00020000); }
__device__ __forceinline__ unsigned umin2(unsigned a, unsigned b) { return a < b ? a : b; }
__device__ __forceinline__ unsigned min4(u32x4 v) { return umin2(umin2(v.x, v.y), umin2(v.z, v.w)); }
__device__ __forceinline__ bool all_nz(unsigned m) { return __builtin_amdgcn_ballot_w64(m == 0u) == 0ull; }
__device__ __forceinline__ u32x4 nz(u32x4 v) {
  v.x = v.x ? v.x : 0x8000u; v.y = v.y ? v.y : 0x8000u; v.z = v.z ? v.z : 0x8000u; v.w = v.w ? v.w : 0x8000u; return v;
}
__device__ __forceinline__ unsigned pack2(float a, float b) { f32x2 f = {a, b}; return __builtin_bit_cast(unsigned, __builtin_convertvector(f, bf16x2)); }
__device__ __forceinline__ float lo16(unsigned u) { return __builtin_bit_cast(float, u << 16); }
__device__ __forceinline__ float hi16(unsigned u) { return __builtin_bit_cast(float, u & 0xffff0000u); }
__device__ __forceinline__ float silu(float y) {
  float c = __builtin_amdgcn_fmed3f(y, -60.f, 60.f);
  float e = __builtin_amdgcn_exp2f(-1.4426950408889634f * c);
  return y * __builtin_amdgcn_rcpf(1.f + e);
}
#define CLOBBER() asm volatile("" ::: "memory")
__device__ __forceinline__ void st128(u32x4 v, rsrc_t r, unsigned off) { __builtin_amdgcn_raw_buffer_store_b128(v, r, off, 0, SYS); }
__device__ __forceinline__ void wait_until(u64 t0, unsigned ticks) {
  for (;;) {
    const unsigned el = (unsigned)(now() - t0);
    if (el >= ticks) break;
    if (ticks - el > 60u) __builtin_amdgcn_s_sleep(16); else __builtin_amdgcn_s_sleep(1);
  }
}
__device__ __forceinline__ unsigned wimg(unsigned row, unsigned c16) { return row * 128u + ((c16 ^ ((row & 2u) | ((row >> 1) & 4u))) << 4); }
__device__ __forceinline__ u32x4 tr_frag(const char* img, unsigned row0, unsigned ni, unsigned lane) {
  const unsigned g = lane >> 4, q = (lane & 15u) >> 2, p = lane & 3u;
  const unsigned r0 = row0 + 8u * g + q;
  s16x4 a = __builtin_amdgcn_ds_read_tr16_b64_v4i16((LDSP s16x4*)(img + wimg(r0, 2u * ni + (p >> 1)) + 8u * (p & 1u)));
  s16x4 b = __builtin_amdgcn_ds_read_tr16_b64_v4i16((LDSP s16x4*)(img + wimg(r0 + 4u, 2u * ni + (p >> 1)) + 8u * (p & 1u)));
  u32x2 ua = __builtin_bit_cast(u32x2, a), ub = __builtin_bit_cast(u32x2, b);
  u32x4 r = {ua.x, ua.y, ub.x, ub.y};
  return r;
}
#define STAMP(i) do {} while (0)

__global__ void __launch_bounds__(512, 2) gemm_rs(Args a) {
  extern __shared__ __attribute__((aligned(16))) char smem[];
  const u64 t0 = now();
  u64 pc0;
  asm volatile("s_getpc_b64 %0" : "=s"(pc0));
  const unsigned b = blockIdx.x;
  const unsigned lane = threadIdx.x & 63u;
  const unsigned wave = __builtin_amdgcn_readfirstlane(threadIdx.x >> 6);
  const unsigned c = 3u - ((b >> 6) & 3u), qrel = ((b >> 3) + 2u) & 7u, j = (b + (TROT ? qrel : 0u)) & 7u;
  const unsigned e6 = 2u + (c * 8u + j) % 6u;
  const unsigned erel = (qrel == 0u && c < 3u) ? e6 : qrel;
  char* p_dst = a.ws[erel]; char* p_my = a.my; const char* p_x = a.x; const char* p_w = a.w; char* p_out = a.out;
  unsigned p_rank = a.rank, p_tx0 = a.tx0, p_ds = a.dslot, opt = a.front, p_to0 = a.to0, p_z = a.zdelay, p_o = a.odelay, p_rotq = a.pad0, p_dso = a.dso, p_pg = a.pgate;
  asm volatile("" : "+s"(p_dst), "+s"(p_my), "+s"(p_x), "+s"(p_w), "+s"(p_out), "+s"(p_rank), "+s"(p_tx0), "+s"(p_ds), "+s"(opt), "+s"(p_to0), "+s"(p_z), "+s"(p_o), "+s"(p_rotq), "+s"(p_dso), "+s"(p_pg));
  const unsigned rank = p_rank & 7u;
  const unsigned rm = rank >> 2, rn = (rank >> 1) & 1u, rk = rank & 1u;
  const unsigned q = rank ^ qrel, rt = j >> 2, tj = 4u * c + (j & 3u);
  const bool xsnd = qrel >= 2u && c == 3u, wsnd = qrel == 0u && c < 3u;
  const unsigned w24 = c * 8u + j;
  const rsrc_t rs_my = mk(p_my, WS_NEED);
  const rsrc_t rs_x = mk(p_x, X_BYTES), rs_w = mk(p_w, W_BYTES);
  STAMP(14);
  const unsigned callv = __builtin_amdgcn_raw_buffer_load_b32(rs_my, CTR_OFF + b * 4u, 0, SYS);
  unsigned call;
  bool dead = false;
  unsigned t_w0 = 80u + ((XSLOTS * p_ds) >> 8), t_w1 = 80u + (((XSLOTS + 256u) * p_ds) >> 8);
  unsigned sink = 0;
  auto rebase = [&]() {
    wait_until(t0, 300u);
    const unsigned an = __builtin_amdgcn_raw_buffer_load_b32(rs_my, CTR_OFF + 4096u + (erel < 2u ? 2u : erel) * 128u + (lane & 7u) * 4u, 0, SYS);
    int d = (int)(an - (unsigned)t0);
    d = (d < -200 || d > 400) ? 1000 : d;
    int v = d * 256;
    #pragma unroll
    for (int o = 1; o < 8; ++o) { const int e = __shfl_xor(d, o); v -= (e < d || (e == d && ((lane ^ o) & 7u) < (lane & 7u))) ? (int)(4u * p_ds) : 0; }
    v = d == 1000 ? -(1 << 20) : v;
    #pragma unroll
    for (int o = 1; o < 8; o <<= 1) { const int e = __shfl_xor(v, o); v = e > v ? e : v; }
    v = __builtin_amdgcn_readfirstlane(v);
    if (v > -(1 << 20)) { t_w0 = (unsigned)((v + (int)(XSLOTS * p_ds)) >> 8); t_w1 = (unsigned)((v + (int)((XSLOTS + 256u) * p_ds)) >> 8); }
  };

  unsigned dbase = 0, wslot = 0;
  char* wpark = smem + wave * 16384u;
  const rsrc_t rs_d = mk(p_dst, WS_NEED);
  auto w_burst = [&]() {
    rebase();
    #pragma unroll
    for (int h = 0; h < 2; ++h) {
      u32x4 wl[4];
      #pragma unroll
      for (int ni = 0; ni < 4; ++ni) wl[ni] = nz(tr_frag(wpark + h * 4096u, 0, ni, lane));
      wait_until(t0, t_w0 + wslot + (h ? (16u * p_ds) >> 8 : 0u));
      #pragma unroll
      for (int ni = 0; ni < 4; ++ni) st128(wl[ni], rs_d, dbase + h * 16384u + ni * 1024u);
    }
    STAMP(2);
  };
  if (__builtin_expect(xsnd, 1)) {
    const unsigned jj = j;
    u32x4 xs[4];
    if ((opt & 16u) && wave != 0u) __builtin_amdgcn_s_sleep(6);
    #pragma unroll
    for (int ks = 0; ks < 4; ++ks)
      xs[ks] = __builtin_amdgcn_raw_buffer_load_b128(rs_x, ((lane & 15u) * 256u + ks * 32u + (lane >> 4) * 8u) * 2u,
                                                     (((q ^ wave) * 256u + (q >> 2) * 128u + jj * 16u) * 256u + (q & 1u) * 128u) * 2u, 0);
    CLOBBER();
    if (wave != 0u) {
      __builtin_amdgcn_s_barrier();
      const unsigned xbase = (unsigned)(now() - t0) - p_tx0;
      if (wave == 7u && lane == 0u) __builtin_amdgcn_raw_buffer_store_b32((unsigned)t0 + xbase, rs_my, CTR_OFF + 4096u + qrel * 128u + j * 4u, 0, SYS);
      t_w0 = xbase + ((XSLOTS * p_ds) >> 8);
      wait_until(t0, xbase + (((wave - (wave > qrel ? 1u : 0u)) * 32u * p_ds) >> 8));
    }
    CLOBBER();
    call = __builtin_amdgcn_readfirstlane(callv) + 1u;
    if (wave != qrel) {
      dbase = (call % 3u) * ROT_BYTES + (qrel - 2u) * SLOT_BYTES + (wave * 8u + jj) * 4096u + lane * 16u;
      #pragma unroll
      for (int ks = 0; ks < 4; ++ks) st128(nz(xs[ks]), rs_d, dbase + ks * 1024u);
    }
    if (wave == 0u) {
      CLOBBER();
      __builtin_amdgcn_s_barrier();
      t_w0 = (unsigned)(now() - t0) + ((XSLOTS * p_ds) >> 8);
    }
    STAMP(0);
    wait_until(t0, t_w0);
  } else if (wsnd) {
    const unsigned d = rank ^ erel, k4 = XBIND ? (w24 & 3u) : w24 / 6u;
    wait_until(t0, p_o);
    u32x4 wl[2][4];
    #pragma unroll
    for (int h = 0; h < 2; ++h)
      #pragma unroll
      for (int i = 0; i < 4; ++i)
        wl[h][i] = __builtin_amdgcn_raw_buffer_load_b128(rs_w, ((i * 8u + (lane >> 3)) * 2048u + (lane & 7u) * 8u + h * 64u) * 2u,
                                                         (((d & 1u) * 128u + k4 * 32u) * 2048u + ((d >> 1) & 1u) * 1024u + wave * 128u) * 2u, 0);
    call = __builtin_amdgcn_readfirstlane(callv) + 1u;
    dbase = (call % 3u) * ROT_BYTES + (erel - 2u) * SLOT_BYTES + 256u * KIB + (wave * 8u + k4) * 4096u + lane * 16u;
    #pragma unroll
    for (int h = 0; h < 2; ++h)
      #pragma unroll
      for (int i = 0; i < 4; ++i) *(LDSP u32x4*)(wpark + h * 4096u + wimg(i * 8u + (lane >> 3), lane & 7u)) = wl[h][i];
    wslot = ((wave * 8u + k4) * 4u * p_ds) >> 8;
  } else {
    if ((opt & 1u) && (b >> 3) == 16u) {
      const rsrc_t rs_c = mk((const char*)(pc0 & ~255ull), CODE_PF);
      const u32x4 c0 = __builtin_amdgcn_raw_buffer_load_b128(rs_c, wave * 2048u + lane * 16u, 0, 0);
      const u32x4 c1 = __builtin_amdgcn_raw_buffer_load_b128(rs_c, wave * 2048u + 1024u + lane * 16u, 0, 0);
      sink = c0.x ^ c1.x;
    }
    call = __builtin_amdgcn_readfirstlane(callv) + 1u;
  }

  const unsigned rot = call % 3u;
  const bool fin = qrel == 0u || (qrel >= 2u && c < 2u);
  const unsigned omh = rm ^ (qrel >> 2), onh = rn ^ ((qrel >> 1) & 1u);
  const unsigned otj = qrel == 0u ? tj : 4u * ((c == 0u ? 2u : 0u) + (qrel & 1u)) + (j & 3u);
  const unsigned pdst = wave * 1024u + lane * 16u + rot * ROT_BYTES + IN_BYTES + rk * MIB + (((rm * 2u + rt) * 32u + rn * 16u + tj) << 13);
  const unsigned ooff = ((wave * 8u + (lane >> 3)) * 2048u + (lane & 7u) * 8u) * 2u + ((omh * 128u + rt * 64u) * 2048u + onh * 1024u + otj * 64u) * 2u;
  if ((opt & 2u) && wave == 7u) {
    sink ^= __builtin_amdgcn_raw_buffer_load_b32(mk(p_out, Y_BYTES), __builtin_amdgcn_readfirstlane(ooff), 0, 0);
  }
  asm volatile("" :: "v"(sink));
  const bool extra = wave + 2u == qrel;
  const bool local = wave >= 6u || extra;
  f32x4 acc[4][4];
  if (wsnd) { w_burst(); wait_until(t0, t_w1); }
  if (!local) {
    u32x4 xf[4][4], wf[4][4];
    const unsigned sbase = rot * ROT_BYTES + wave * SLOT_BYTES;
    const unsigned xblk = (qrel * 2u + rt) * 16384u, wblk = 256u * KIB + tj * 16384u;
    const unsigned moff = (lane < 32u ? xblk : wblk) + ((lane >> 3) & 3u) * 4096u + 4064u + (lane & 7u) * 4u;
    auto poll = [&](u64 need) -> u64 {
      for (;;) {
        const u64 miss = __builtin_amdgcn_ballot_w64(__builtin_amdgcn_raw_buffer_load_b32(rs_my, moff, sbase, SYS) == 0u);
        if ((miss & need) == 0ull) return miss;
        if (now() - t0 > DEADLINE) { dead = true; return 0ull; }
        __builtin_amdgcn_s_sleep(3); CLOBBER();
      }
    };
    bool xok = false;
    for (;;) {
      unsigned m = ~0u;
      if (!xok) {
        const u64 miss = poll(0xffffffffull);
        STAMP(3);
        #pragma unroll
        for (int i = 0; i < 16; ++i) xf[i >> 2][i & 3] = __builtin_amdgcn_raw_buffer_load_b128(rs_my, lane * 16u + (i & 3) * 1024u, sbase + xblk + (i >> 2) * 4096u, SYS);
        if (miss != 0ull) {
          #pragma unroll
          for (int i = 0; i < 16; ++i) m = umin2(m, min4(xf[i >> 2][i & 3]));
          if (!all_nz(m)) { __builtin_amdgcn_s_sleep(4); CLOBBER(); continue; }
          xok = true;
          STAMP(4);
          poll(0xffffffff00000000ull);
        }
        STAMP(5);
      }
      CLOBBER();
      #pragma unroll
      for (int i = 0; i < 16; ++i) wf[i >> 2][i & 3] = __builtin_amdgcn_raw_buffer_load_b128(rs_my, lane * 16u + (i & 3) * 1024u, sbase + wblk + (i >> 2) * 4096u, SYS);
      unsigned mx = ~0u;
      #pragma unroll
      for (int ks = 0; ks < 4; ++ks)
        #pragma unroll
        for (int ni = 0; ni < 4; ++ni) {
          __builtin_amdgcn_sched_barrier(0);
          #pragma unroll
          for (int mi = 0; mi < 4; ++mi)
            acc[mi][ni] = __builtin_amdgcn_mfma_f32_16x16x32_bf16(__builtin_bit_cast(bf16x8, wf[ks][ni]), __builtin_bit_cast(bf16x8, xf[mi][ks]),
                                                                  ks ? acc[mi][ni] : (f32x4){0.f, 0.f, 0.f, 0.f}, 0, 0, 0);
          m = umin2(m, min4(wf[ks][ni]));
          mx = umin2(mx, min4(xf[ni][ks]));
        }
      xok = all_nz(mx);
      if (xok && all_nz(m)) break;
      if (dead || now() - t0 > DEADLINE) { dead = true; break; }
    }
    if (dead) acc[0][0] = acc[1][0] = acc[2][0] = acc[3][0] = (f32x4){__builtin_nanf(""), __builtin_nanf(""), __builtin_nanf(""), __builtin_nanf("")};
    STAMP(6);
  } else {
    #pragma unroll
    for (int i = 0; i < 16; ++i) acc[i >> 2][i & 3] = (f32x4){0.f, 0.f, 0.f, 0.f};
    wait_until(t0, p_o);
    const unsigned lrow = extra ? rank * 256u + omh * 128u + rt * 64u : q * 256u + rm * 128u + rt * 64u;
    const unsigned lcol = extra ? onh * 1024u + otj * 64u : rn * 1024u + tj * 64u;
    const unsigned k0 = extra ? 0u : (wave - 6u) * 128u, k1 = extra ? (fin ? 256u : 0u) : k0 + 128u;
    char* img = smem + wave * 16384u;
    const bool zer = !extra && qrel != 0u && (c == 1u || c == 2u);
    #pragma unroll 1
    for (unsigned ph = 0; ph < 2u; ++ph) {
      if ((ph ^ wave) & 1u) {
        if (zer) {
          const unsigned zi = (((qrel - 1u) * 2u + (c - 1u)) * 8u + j) * 2u + (wave - 6u);
          const unsigned zb = ((call + 2u) % 3u) * ROT_BYTES + lane * 16u;
          const u32x4 z = {0u, 0u, 0u, 0u};
          unsigned tz = (unsigned)(now() - t0) + (zi & 15u);
          #pragma unroll 1
          for (unsigned i = zi; i < ROT_BYTES / 1024u; i += 224u) {
            wait_until(t0, tz); tz += p_z;
            __builtin_amdgcn_raw_buffer_store_b128(z, rs_my, zb + i * 1024u, 0, 16);
          }
        }
      } else {
        #pragma unroll 1
        for (unsigned k = k0; k < k1; k += 128u) {
          u32x4 wr[16], xf[4][4];
          #pragma unroll
          for (int i = 0; i < 16; ++i)
            wr[i] = __builtin_amdgcn_raw_buffer_load_b128(rs_w, ((lane >> 3) * 2048u + (lane & 7u) * 8u) * 2u, ((k + i * 8u) * 2048u + lcol) * 2u, 0);
          #pragma unroll
          for (int mi = 0; mi < 4; ++mi)
            #pragma unroll
            for (int ks = 0; ks < 4; ++ks)
              xf[mi][ks] = __builtin_amdgcn_raw_buffer_load_b128(rs_x, ((lane & 15u) * 256u + ks * 32u + (lane >> 4) * 8u) * 2u, ((lrow + mi * 16u) * 256u + k) * 2u, 0);
          __builtin_amdgcn_sched_barrier(0);
          #pragma unroll
          for (int i = 0; i < 16; ++i) *(LDSP u32x4*)(img + wimg(i * 8u + (lane >> 3), lane & 7u)) = wr[i];
          #pragma unroll
          for (int ks = 0; ks < 4; ++ks)
            #pragma unroll
            for (int ni = 0; ni < 4; ++ni) {
              const u32x4 wf = tr_frag(img, ks * 32u, ni, lane);
              #pragma unroll
              for (int mi = 0; mi < 4; ++mi)
                acc[mi][ni] = __builtin_amdgcn_mfma_f32_16x16x32_bf16(__builtin_bit_cast(bf16x8, wf), __builtin_bit_cast(bf16x8, xf[mi][ks]), acc[mi][ni], 0, 0, 0);
            }
          CLOBBER();
        }
      }
    }
    if (wave == 7u && !wsnd) { rebase(); if (lane == 0u) *(LDSP unsigned*)(smem + LDS_TW) = t_w1 + (unsigned)t0; }
  }
  {
    char* slab = smem + wave * 16384u;
    #pragma unroll
    for (int mi = 0; mi < 4; ++mi)
      #pragma unroll
      for (int ni = 0; ni < 4; ++ni) {
        const unsigned nq = ni * 4u + (lane >> 4);
        *(LDSP f32x4*)(slab + (((mi * 16u + nq) * 16u + ((lane & 15u) ^ (nq & 14u))) << 4)) = acc[mi][ni];
      }
  }
  STAMP(7);
  asm volatile("s_waitcnt lgkmcnt(0)" ::: "memory");
  __builtin_amdgcn_s_barrier();
  CLOBBER();
  if (threadIdx.x == 0) __builtin_amdgcn_raw_buffer_store_b32(call, rs_my, CTR_OFF + b * 4u, 0, 0);
  t_w1 = __builtin_amdgcn_readfirstlane(*(LDSP unsigned*)(smem + LDS_TW)) - (unsigned)t0;
  f32x4 s0, s1, e0, e1;
  {
    const unsigned m = wave * 8u + (lane >> 3), o = lane & 7u;
    const unsigned ad = (((m >> 4) * 16u + 2u * o) * 16u + ((m & 15u) ^ (2u * o))) << 4;
    const unsigned sx = (qrel - 2u) & 7u;
    e0 = *(LDSP f32x4*)(smem + sx * 16384u + ad);
    e1 = *(LDSP f32x4*)(smem + sx * 16384u + ad + 256u);
    s0 = *(LDSP f32x4*)(smem + (sx ^ 1u) * 16384u + ad);
    s1 = *(LDSP f32x4*)(smem + (sx ^ 1u) * 16384u + ad + 256u);
    #pragma unroll
    for (int s = 2; s < 8; ++s) {
      s0 += *(LDSP f32x4*)(smem + (sx ^ s) * 16384u + ad);
      s1 += *(LDSP f32x4*)(smem + (sx ^ s) * 16384u + ad + 256u);
    }
    if (qrel < 2u) { s0 += e0; s1 += e1; e0 = s0; e1 = s1; }
  }
  STAMP(8);
  const unsigned strip = wave * 1024u + lane * 16u;
  if (qrel != 0u) {
    u32x4 p = {pack2(s0.x, s0.y), pack2(s0.z, s0.w), pack2(s1.x, s1.y), pack2(s1.z, s1.w)};
    const unsigned oslot = c * 8u + ((((j & 3u) * 2u + rt) + qrel * p_rotq) & 7u);
    const unsigned pg = qrel == 1u ? p_pg : 0u;
    wait_until(t0, t_w1 + p_to0 - pg + (((oslot * 8u + wave) * (p_dso + ((opt & 256u) ? 0u : pg))) >> 8));
    st128(nz(p), rs_d, pdst);
    STAMP(9);
  }
  if (fin) {
    const unsigned tbase = rot * ROT_BYTES + IN_BYTES + (((omh * 2u + rt) * 32u + onh * 16u + otj) << 13);
    const unsigned pa = tbase + (rk ^ 1u) * MIB, pb = tbase + (qrel != 0u ? rk : (rk ^ 1u)) * MIB;
    u32x4 u, v;
    dead = false;
    for (unsigned it = 1;; ++it) {
      u = __builtin_amdgcn_raw_buffer_load_b128(rs_my, strip, pa, SYS);
      v = __builtin_amdgcn_raw_buffer_load_b128(rs_my, strip, pb, SYS);
      if (all_nz(umin2(min4(u), min4(v)))) break;
      if ((it & 63u) == 0u && now() - t0 > DEADLINE) { dead = true; break; }
      if (!(opt & 32u)) __builtin_amdgcn_s_sleep(2);
      CLOBBER();
    }
    STAMP(10);
    if (qrel == 0u) v = (u32x4){0u, 0u, 0u, 0u};
    float y0 = e0.x + lo16(u.x) + lo16(v.x), y1 = e0.y + hi16(u.x) + hi16(v.x), y2 = e0.z + lo16(u.y) + lo16(v.y), y3 = e0.w + hi16(u.y) + hi16(v.y);
    float y4 = e1.x + lo16(u.z) + lo16(v.z), y5 = e1.y + hi16(u.z) + hi16(v.z), y6 = e1.z + lo16(u.w) + lo16(v.w), y7 = e1.w + hi16(u.w) + hi16(v.w);
    u32x4 o = {pack2(silu(y0), silu(y1)), pack2(silu(y2), silu(y3)), pack2(silu(y4), silu(y5)), pack2(silu(y6), silu(y7))};
    if (dead) o = (u32x4){0x7fc07fc0u, 0x7fc07fc0u, 0x7fc07fc0u, 0x7fc07fc0u};
    __builtin_amdgcn_raw_buffer_store_b128(o, mk(p_out, Y_BYTES), ooff, 0, 0);
  }
}

static const int kDefaults[10] = {10, 435, 16, 0, 12, 120, TROT ? 0 : 1, 0, 440, 0};

extern "C" void kernel_launch(int rank, int nranks, ncclComm_t comm, hipStream_t stream,
                              void* const* d_in, const int* in_sizes, int n_in,
                              void* d_out, int out_size, const int* in_dtypes, int out_dtype,
                              void* const* d_out_all, void* const* d_ws_all, size_t ws_bytes) {
  if (nranks != 8 || ws_bytes < WS_NEED || n_in < 2) { fprintf(stderr, "unexpected configuration\n"); return; }
  static bool init = false;
  const int lds = LDS_BYTES;
  if (!init) { (void)hipFuncSetAttribute((const void*)gemm_rs, hipFuncAttributeMaxDynamicSharedMemorySize, lds); init = true; }
  Args a;
  for (int p = 0; p < 8; ++p) a.ws[p] = (char*)d_ws_all[(rank ^ p) & 7];
  a.my = a.ws[0]; a.x = (const char*)d_in[0]; a.w = (const char*)d_in[1]; a.out = (char*)d_out; a.rank = rank;
  const int* cf = kDefaults;
  a.tx0 = cf[0]; a.dslot = cf[1]; a.front = cf[2]; a.to0 = cf[3]; a.zdelay = cf[4]; a.odelay = cf[5]; a.pad0 = cf[6]; a.pad1 = cf[7]; a.dso = cf[8]; a.pgate = cf[9];
  gemm_rs<<<256, 512, lds, stream>>>(a);
}
```

```cpp
#include <hip/hip_runtime.h>
#include <rccl/rccl.h>
#include <cstdio>

typedef unsigned u32x4 __attribute__((ext_vector_type(4)));
typedef unsigned u32x2 __attribute__((ext_vector_type(2)));
typedef short s16x4 __attribute__((ext_vector_type(4)));
typedef __bf16 bf16x8 __attribute__((ext_vector_type(8)));
typedef __bf16 bf16x2 __attribute__((ext_vector_type(2)));
typedef float f32x2 __attribute__((ext_vector_type(2)));
typedef float f32x4 __attribute__((ext_vector_type(4)));
typedef unsigned long long u64;
typedef __amdgpu_buffer_rsrc_t rsrc_t;
#define LDSP __attribute__((address_space(3)))
#define CLOBBER() asm volatile("" ::: "memory")

constexpr unsigned KIB = 1024u, MIB = 1024u * 1024u;
constexpr unsigned SLOT_BYTES = 512 * KIB;
constexpr unsigned IN_BYTES = 6 * SLOT_BYTES;
constexpr unsigned PART_BYTES = 2 * MIB;
constexpr unsigned ROT_BYTES = IN_BYTES + PART_BYTES;
constexpr unsigned CTR_OFF = 3 * ROT_BYTES;
constexpr unsigned ANC_OFF = CTR_OFF + 4096u;
constexpr unsigned WS_NEED = CTR_OFF + 64 * KIB;
constexpr unsigned X_BYTES = 2048u * 256u * 2u, W_BYTES = 256u * 2048u * 2u, Y_BYTES = 256u * 2048u * 2u;
constexpr u64 DEADLINE = 50000000ull;
#define P_DS 435
#define P_DSO 440
#define P_LEAD 15
constexpr unsigned DS_IN = P_DS, DS_OUT = P_DSO;
constexpr unsigned XSLOTS = 224u, WSLOTS = 256u;
constexpr unsigned LEAD = P_LEAD;
constexpr unsigned T_LOCAL = 120u;
constexpr unsigned Z_STEP = 12u;
constexpr int SYS = 17, DEV = 16;
constexpr int LDS_TW = 128 * 1024, LDS_BYTES = LDS_TW + 64;

struct Args {
  const char* x; char* my; const char* w; char* out;
  long long rank;
  char* ws[8];
};

__device__ __forceinline__ u64 now() { return __builtin_amdgcn_s_memrealtime(); }
__device__ __forceinline__ rsrc_t mk(const void* p, unsigned bytes) { return __builtin_amdgcn_make_buffer_rsrc((void*)p, 0, bytes, 0x00020000); }
__device__ __forceinline__ unsigned umin2(unsigned a, unsigned b) { return a < b ? a : b; }
__device__ __forceinline__ unsigned min4(u32x4 v) { return umin2(umin2(v.x, v.y), umin2(v.z, v.w)); }
__device__ __forceinline__ bool all_nz(unsigned m) { return __builtin_amdgcn_ballot_w64(m == 0u) == 0ull; }
__device__ __forceinline__ u32x4 nz(u32x4 v) {
  v.x = v.x ? v.x : 0x8000u; v.y = v.y ? v.y : 0x8000u; v.z = v.z ? v.z : 0x8000u; v.w = v.w ? v.w : 0x8000u; return v;
}
__device__ __forceinline__ unsigned pack2(float a, float b) { f32x2 f = {a, b}; return __builtin_bit_cast(unsigned, __builtin_convertvector(f, bf16x2)); }
__device__ __forceinline__ float lo16(unsigned u) { return __builtin_bit_cast(float, u << 16); }
__device__ __forceinline__ float hi16(unsigned u) { return __builtin_bit_cast(float, u & 0xffff0000u); }
__device__ __forceinline__ float silu(float y) {
  const float c = __builtin_amdgcn_fmed3f(y, -60.f, 60.f);
  const float e = __builtin_amdgcn_exp2f(-1.4426950408889634f * c);
  return y * __builtin_amdgcn_rcpf(1.f + e);
}
template <int AUX> __device__ __forceinline__ void st128(u32x4 v, rsrc_t r, unsigned off) { __builtin_amdgcn_raw_buffer_store_b128(v, r, off, 0, AUX); }
template <int AUX> __device__ __forceinline__ u32x4 ld128(rsrc_t r, unsigned voff, unsigned soff) { return __builtin_amdgcn_raw_buffer_load_b128(r, voff, soff, AUX); }
__device__ __forceinline__ void wait_until(u64 t0, unsigned ticks) {
  for (;;) {
    const unsigned el = (unsigned)(now() - t0);
    if (el >= ticks) break;
    if (ticks - el > 60u) __builtin_amdgcn_s_sleep(16); else __builtin_amdgcn_s_sleep(1);
  }
}
__device__ __forceinline__ f32x4 mfma(u32x4 wf, u32x4 xf, f32x4 c) {
  return __builtin_amdgcn_mfma_f32_16x16x32_bf16(__builtin_bit_cast(bf16x8, wf), __builtin_bit_cast(bf16x8, xf), c, 0, 0, 0);
}
__device__ __forceinline__ unsigned wimg(unsigned row, unsigned c16) { return row * 128u + ((c16 ^ ((row & 2u) | ((row >> 1) & 4u))) << 4); }
__device__ __forceinline__ u32x4 tr_frag(const char* img, unsigned row0, unsigned ni, unsigned lane) {
  const unsigned g = lane >> 4, q = (lane & 15u) >> 2, p = lane & 3u;
  const unsigned r0 = row0 + 8u * g + q;
  const s16x4 a = __builtin_amdgcn_ds_read_tr16_b64_v4i16((LDSP s16x4*)(img + wimg(r0, 2u * ni + (p >> 1)) + 8u * (p & 1u)));
  const s16x4 b = __builtin_amdgcn_ds_read_tr16_b64_v4i16((LDSP s16x4*)(img + wimg(r0 + 4u, 2u * ni + (p >> 1)) + 8u * (p & 1u)));
  const u32x2 ua = __builtin_bit_cast(u32x2, a), ub = __builtin_bit_cast(u32x2, b);
  return (u32x4){ua.x, ua.y, ub.x, ub.y};
}

__global__ void __launch_bounds__(512, 2) gemm_rs(Args a) {
  extern __shared__ __attribute__((aligned(16))) char smem[];
  const unsigned b = blockIdx.x;
  char* p_dst = a.ws[(b >> 3) & 7u]; char* p_my = a.my; const char* p_x = a.x; const char* p_w = a.w; char* p_out = a.out;
  unsigned p_rank = (unsigned)a.rank;
  const u64 t0 = now();
  asm volatile("" : "+s"(p_dst), "+s"(p_my), "+s"(p_x), "+s"(p_w), "+s"(p_out), "+s"(p_rank));
  const unsigned lane = threadIdx.x & 63u;
  const unsigned wave = __builtin_amdgcn_readfirstlane(threadIdx.x >> 6);
  const unsigned c = 3u - ((b >> 6) & 3u), qrel = ((b >> 3) + 2u) & 7u, j = (b + qrel) & 7u;
  const unsigned rt = j >> 2, tj = 4u * c + (j & 3u), w24 = c * 8u + j;
  const bool xsnd = b < 48u, wsnd = qrel == 0u && c < 3u;
  const unsigned erel = wsnd ? 2u + w24 % 6u : qrel;
  const unsigned rank = p_rank & 7u;
  const unsigned rm = rank >> 2, rn = (rank >> 1) & 1u, rk = rank & 1u, q = rank ^ qrel;
  const rsrc_t rs_my = mk(p_my, WS_NEED), rs_d = mk(p_dst, WS_NEED), rs_x = mk(p_x, X_BYTES), rs_w = mk(p_w, W_BYTES);
  const unsigned callv = __builtin_amdgcn_raw_buffer_load_b32(rs_my, CTR_OFF + b * 4u, 0, SYS);
  unsigned call;
  unsigned t_w0 = 80u + ((XSLOTS * DS_IN) >> 8);
  auto rebase = [&]() {
    wait_until(t0, 300u);
    const unsigned an = __builtin_amdgcn_raw_buffer_load_b32(rs_my, ANC_OFF + (erel < 2u ? 2u : erel) * 128u + (lane & 7u) * 4u, 0, SYS);
    int d = (int)(an - (unsigned)t0);
    d = (d < -200 || d > 400) ? 1000 : d;
    int v = d * 256;
    #pragma unroll
    for (int o = 1; o < 8; ++o) { const int e = __shfl_xor(d, o); v -= (e < d || (e == d && ((lane ^ o) & 7u) < (lane & 7u))) ? (int)(4u * DS_IN) : 0; }
    v = d == 1000 ? -(1 << 20) : v;
    #pragma unroll
    for (int o = 1; o < 8; o <<= 1) { const int e = __shfl_xor(v, o); v = e > v ? e : v; }
    v = __builtin_amdgcn_readfirstlane(v);
    if (v > -(1 << 20)) t_w0 = (unsigned)((v + (int)(XSLOTS * DS_IN)) >> 8);
  };

  if (__builtin_expect(xsnd, 1)) {
    u32x4 xs[4];
    if (wave != 0u) __builtin_amdgcn_s_sleep(6);
    #pragma unroll
    for (int ks = 0; ks < 4; ++ks)
      xs[ks] = ld128<0>(rs_x, ((lane & 15u) * 256u + ks * 32u + (lane >> 4) * 8u) * 2u, (((q ^ wave) * 256u + (q >> 2) * 128u + j * 16u) * 256u + (q & 1u) * 128u) * 2u);
    CLOBBER();
    if (wave != 0u) {
      __builtin_amdgcn_s_barrier();
      const unsigned xbase = (unsigned)(now() - t0) - LEAD;
      if (wave == 7u && lane == 0u) __builtin_amdgcn_raw_buffer_store_b32((unsigned)t0 + xbase, rs_my, ANC_OFF + qrel * 128u + j * 4u, 0, SYS);
      t_w0 = xbase + ((XSLOTS * DS_IN) >> 8);
      wait_until(t0, xbase + (((wave - (wave > qrel ? 1u : 0u)) * 32u * DS_IN) >> 8));
    }
    CLOBBER();
    call = __builtin_amdgcn_readfirstlane(callv) + 1u;
    if (wave != qrel) {
      const unsigned dbase = (call % 3u) * ROT_BYTES + (qrel - 2u) * SLOT_BYTES + (wave * 8u + j) * 4096u + lane * 16u;
      #pragma unroll
      for (int ks = 0; ks < 4; ++ks) st128<SYS>(nz(xs[ks]), rs_d, dbase + ks * 1024u);
    }
    if (wave == 0u) {
      CLOBBER();
      __builtin_amdgcn_s_barrier();
      t_w0 = (unsigned)(now() - t0) + ((XSLOTS * DS_IN) >> 8);
    }
    wait_until(t0, t_w0);
  } else if (wsnd) {
    const unsigned d = rank ^ erel, k4 = w24 / 6u;
    const rsrc_t rs_e = mk(a.ws[(erel - 2u) & 7u], WS_NEED);
    wait_until(t0, T_LOCAL);
    u32x4 wl[2][4];
    #pragma unroll
    for (int h = 0; h < 2; ++h)
      #pragma unroll
      for (int i = 0; i < 4; ++i)
        wl[h][i] = ld128<0>(rs_w, ((lane >> 3) * 2048u + (lane & 7u) * 8u + h * 64u) * 2u,
                            (((d & 1u) * 128u + k4 * 32u + i * 8u) * 2048u + ((d >> 1) & 1u) * 1024u + wave * 128u) * 2u);
    call = __builtin_amdgcn_readfirstlane(callv) + 1u;
    const unsigned dbase = (call % 3u) * ROT_BYTES + (erel - 2u) * SLOT_BYTES + 256u * KIB + (wave * 8u + k4) * 4096u + lane * 16u;
    char* img = smem + wave * 16384u;
    #pragma unroll
    for (int h = 0; h < 2; ++h)
      #pragma unroll
      for (int i = 0; i < 4; ++i) *(LDSP u32x4*)(img + h * 4096u + wimg(i * 8u + (lane >> 3), lane & 7u)) = wl[h][i];
    #pragma unroll
    for (int h = 0; h < 2; ++h)
      #pragma unroll
      for (int ni = 0; ni < 4; ++ni) wl[h][ni] = nz(tr_frag(img + h * 4096u, 0, ni, lane));
    rebase();
    #pragma unroll
    for (int h = 0; h < 2; ++h) {
      wait_until(t0, t_w0 + (((wave * 8u + k4 + h * 4u) * 4u * DS_IN) >> 8));
      #pragma unroll
      for (int ni = 0; ni < 4; ++ni) st128<SYS>(wl[h][ni], rs_e, dbase + h * 16384u + ni * 1024u);
    }
    wait_until(t0, t_w0 + ((WSLOTS * DS_IN) >> 8));
  } else {
    call = __builtin_amdgcn_readfirstlane(callv) + 1u;
  }

  const unsigned rot = call % 3u;
  const bool fin = qrel == 0u || (qrel >= 2u && c < 2u);
  const unsigned omh = rm ^ (qrel >> 2), onh = rn ^ ((qrel >> 1) & 1u);
  const unsigned otj = qrel == 0u ? tj : 4u * ((c == 0u ? 2u : 0u) + (qrel & 1u)) + (j & 3u);
  const bool extra = wave + 2u == qrel;
  f32x4 acc[4][4];
  if (wave < 6u && !extra) {
    u32x4 xf[4][4], wf[4][4];
    bool dead = false;
    const unsigned sbase = rot * ROT_BYTES + wave * SLOT_BYTES;
    const unsigned xblk = (qrel * 2u + rt) * 16384u, wblk = 256u * KIB + tj * 16384u;
    const unsigned moff = (lane < 32u ? xblk : wblk) + ((lane >> 3) & 3u) * 4096u + 4064u + (lane & 7u) * 4u;
    auto poll = [&](u64 need) -> u64 {
      for (;;) {
        const u64 miss = __builtin_amdgcn_ballot_w64(__builtin_amdgcn_raw_buffer_load_b32(rs_my, moff, sbase, SYS) == 0u);
        if ((miss & need) == 0ull) return miss;
        if (now() - t0 > DEADLINE) { dead = true; return 0ull; }
        __builtin_amdgcn_s_sleep(3); CLOBBER();
      }
    };
    bool xok = false;
    for (;;) {
      unsigned m = ~0u;
      if (!xok) {
        const u64 miss = poll(0xffffffffull);
        #pragma unroll
        for (int i = 0; i < 16; ++i) xf[i >> 2][i & 3] = ld128<SYS>(rs_my, lane * 16u + (i & 3) * 1024u, sbase + xblk + (i >> 2) * 4096u);
        if (miss != 0ull) {
          #pragma unroll
          for (int i = 0; i < 16; ++i) m = umin2(m, min4(xf[i >> 2][i & 3]));
          if (!all_nz(m) && now() - t0 <= DEADLINE) { __builtin_amdgcn_s_sleep(4); CLOBBER(); continue; }
          poll(0xffffffff00000000ull);
        }
      }
      CLOBBER();
      #pragma unroll
      for (int i = 0; i < 16; ++i) wf[i >> 2][i & 3] = ld128<SYS>(rs_my, lane * 16u + (i & 3) * 1024u, sbase + wblk + (i >> 2) * 4096u);
      unsigned mx = ~0u;
      #pragma unroll
      for (int ks = 0; ks < 4; ++ks)
        #pragma unroll
        for (int ni = 0; ni < 4; ++ni) {
          __builtin_amdgcn_sched_barrier(0);
          #pragma unroll
          for (int mi = 0; mi < 4; ++mi) acc[mi][ni] = mfma(wf[ks][ni], xf[mi][ks], ks ? acc[mi][ni] : (f32x4){0.f, 0.f, 0.f, 0.f});
          m = umin2(m, min4(wf[ks][ni]));
          mx = umin2(mx, min4(xf[ni][ks]));
        }
      xok = all_nz(mx);
      if (xok && all_nz(m)) break;
      if (dead || now() - t0 > DEADLINE) { dead = true; break; }
    }
    if (dead) acc[0][0] = acc[1][0] = acc[2][0] = acc[3][0] = (f32x4){__builtin_nanf(""), __builtin_nanf(""), __builtin_nanf(""), __builtin_nanf("")};
  } else {
    #pragma unroll
    for (int i = 0; i < 16; ++i) acc[i >> 2][i & 3] = (f32x4){0.f, 0.f, 0.f, 0.f};
    wait_until(t0, T_LOCAL);
    const unsigned lrow = extra ? rank * 256u + omh * 128u + rt * 64u : q * 256u + rm * 128u + rt * 64u;
    const unsigned lcol = extra ? onh * 1024u + otj * 64u : rn * 1024u + tj * 64u;
    const unsigned k0 = extra ? 0u : (wave - 6u) * 128u, k1 = extra ? (fin ? 256u : 0u) : k0 + 128u;
    char* img = smem + wave * 16384u;
    const bool zer = !extra && qrel != 0u && (c == 1u || c == 2u);
    #pragma unroll 1
    for (unsigned ph = 0; ph < 2u; ++ph) {
      if ((ph ^ wave) & 1u) {
        if (zer) {
          const unsigned zi = (((qrel - 1u) * 2u + (c - 1u)) * 8u + j) * 2u + (wave - 6u);
          const unsigned zb = ((call + 2u) % 3u) * ROT_BYTES + lane * 16u;
          unsigned tz = (unsigned)(now() - t0) + (zi & 15u);
          #pragma unroll 1
          for (unsigned i = zi; i < ROT_BYTES / 1024u; i += 224u) {
            wait_until(t0, tz); tz += Z_STEP;
            st128<DEV>((u32x4){0u, 0u, 0u, 0u}, rs_my, zb + i * 1024u);
          }
        }
      } else {
        #pragma unroll 1
        for (unsigned k = k0; k < k1; k += 128u) {
          u32x4 wr[16], xf[4][4];
          #pragma unroll
          for (int i = 0; i < 16; ++i) wr[i] = ld128<0>(rs_w, ((lane >> 3) * 2048u + (lane & 7u) * 8u) * 2u, ((k + i * 8u) * 2048u + lcol) * 2u);
          #pragma unroll
          for (int mi = 0; mi < 4; ++mi)
            #pragma unroll
            for (int ks = 0; ks < 4; ++ks)
              xf[mi][ks] = ld128<0>(rs_x, ((lane & 15u) * 256u + ks * 32u + (lane >> 4) * 8u) * 2u, ((lrow + mi * 16u) * 256u + k) * 2u);
          __builtin_amdgcn_sched_barrier(0);
          #pragma unroll
          for (int i = 0; i < 16; ++i) *(LDSP u32x4*)(img + wimg(i * 8u + (lane >> 3), lane & 7u)) = wr[i];
          #pragma unroll
          for (int ks = 0; ks < 4; ++ks)
            #pragma unroll
            for (int ni = 0; ni < 4; ++ni) {
              const u32x4 wf = tr_frag(img, ks * 32u, ni, lane);
              #pragma unroll
              for (int mi = 0; mi < 4; ++mi) acc[mi][ni] = mfma(wf, xf[mi][ks], acc[mi][ni]);
            }
          CLOBBER();
        }
      }
    }
    if (wave == 7u && qrel != 0u) { rebase(); if (lane == 0u) *(LDSP unsigned*)(smem + LDS_TW) = t_w0 + (unsigned)t0; }
  }
  {
    char* slab = smem + wave * 16384u;
    #pragma unroll
    for (int mi = 0; mi < 4; ++mi)
      #pragma unroll
      for (int ni = 0; ni < 4; ++ni) {
        const unsigned nq = ni * 4u + (lane >> 4);
        *(LDSP f32x4*)(slab + (((mi * 16u + nq) * 16u + ((lane & 15u) ^ (nq & 14u))) << 4)) = acc[mi][ni];
      }
  }
  asm volatile("s_waitcnt lgkmcnt(0)" ::: "memory");
  __builtin_amdgcn_s_barrier();
  CLOBBER();
  if (threadIdx.x == 0) __builtin_amdgcn_raw_buffer_store_b32(call, rs_my, CTR_OFF + b * 4u, 0, 0);
  f32x4 s0, s1, e0, e1;
  {
    const unsigned m = wave * 8u + (lane >> 3), o = lane & 7u;
    const unsigned ad = (((m >> 4) * 16u + 2u * o) * 16u + ((m & 15u) ^ (2u * o))) << 4;
    const unsigned sx = (qrel - 2u) & 7u;
    e0 = *(LDSP f32x4*)(smem + sx * 16384u + ad);
    e1 = *(LDSP f32x4*)(smem + sx * 16384u + ad + 256u);
    s0 = *(LDSP f32x4*)(smem + (sx ^ 1u) * 16384u + ad);
    s1 = *(LDSP f32x4*)(smem + (sx ^ 1u) * 16384u + ad + 256u);
    #pragma unroll
    for (int s = 2; s < 8; ++s) {
      s0 += *(LDSP f32x4*)(smem + (sx ^ s) * 16384u + ad);
      s1 += *(LDSP f32x4*)(smem + (sx ^ s) * 16384u + ad + 256u);
    }
    if (qrel < 2u) { s0 += e0; s1 += e1; e0 = s0; e1 = s1; }
  }
  const unsigned strip = wave * 1024u + lane * 16u;
  if (qrel != 0u) {
    const u32x4 p = {pack2(s0.x, s0.y), pack2(s0.z, s0.w), pack2(s1.x, s1.y), pack2(s1.z, s1.w)};
    const unsigned tb = __builtin_amdgcn_readfirstlane(*(LDSP unsigned*)(smem + LDS_TW)) - (unsigned)t0 + (qrel == 1u ? 0u : (WSLOTS * DS_IN) >> 8);
    wait_until(t0, tb + ((((tj * 2u + rt) * 8u + wave) * DS_OUT) >> 8));
    st128<SYS>(nz(p), rs_d, strip + rot * ROT_BYTES + IN_BYTES + rk * MIB + (((rm * 2u + rt) * 32u + rn * 16u + tj) << 13));
  }
  if (fin) {
    const unsigned tbase = rot * ROT_BYTES + IN_BYTES + (((omh * 2u + rt) * 32u + onh * 16u + otj) << 13);
    const unsigned pa = tbase + (rk ^ 1u) * MIB, pb = tbase + (qrel != 0u ? rk : (rk ^ 1u)) * MIB;
    u32x4 u, v;
    bool dead = false;
    for (unsigned it = 1;; ++it) {
      u = ld128<SYS>(rs_my, strip, pa);
      v = ld128<SYS>(rs_my, strip, pb);
      if (all_nz(umin2(min4(u), min4(v)))) break;
      if ((it & 63u) == 0u && now() - t0 > DEADLINE) { dead = true; break; }
      __builtin_amdgcn_s_sleep(1);
      CLOBBER();
    }
    if (qrel == 0u) v = (u32x4){0u, 0u, 0u, 0u};
    const float y0 = e0.x + lo16(u.x) + lo16(v.x), y1 = e0.y + hi16(u.x) + hi16(v.x), y2 = e0.z + lo16(u.y) + lo16(v.y), y3 = e0.w + hi16(u.y) + hi16(v.y);
    const float y4 = e1.x + lo16(u.z) + lo16(v.z), y5 = e1.y + hi16(u.z) + hi16(v.z), y6 = e1.z + lo16(u.w) + lo16(v.w), y7 = e1.w + hi16(u.w) + hi16(v.w);
    u32x4 o = {pack2(silu(y0), silu(y1)), pack2(silu(y2), silu(y3)), pack2(silu(y4), silu(y5)), pack2(silu(y6), silu(y7))};
    if (dead) o = (u32x4){0x7fc07fc0u, 0x7fc07fc0u, 0x7fc07fc0u, 0x7fc07fc0u};
    const unsigned ooff = ((omh * 128u + rt * 64u + wave * 8u + (lane >> 3)) * 2048u + onh * 1024u + otj * 64u + (lane & 7u) * 8u) * 2u;
    if (otj < 12u && qrel != 0u) st128<DEV>(o, mk(p_out, Y_BYTES), ooff);
    else st128<0>(o, mk(p_out, Y_BYTES), ooff);
  }
}

extern "C" void kernel_launch(int rank, int nranks, ncclComm_t comm, hipStream_t stream,
                              void* const* d_in, const int* in_sizes, int n_in,
                              void* d_out, int out_size, const int* in_dtypes, int out_dtype,
                              void* const* d_out_all, void* const* d_ws_all, size_t ws_bytes) {
  (void)comm; (void)in_dtypes; (void)out_dtype; (void)d_out_all;
  if (nranks != 8 || (unsigned)rank >= 8u || n_in < 2 || ws_bytes < WS_NEED || (size_t)in_sizes[0] * 2 != X_BYTES ||
      (size_t)in_sizes[1] * 2 != W_BYTES || (size_t)out_size * 2 != Y_BYTES) {
    fprintf(stderr, "kernel_launch: unexpected configuration\n");
    return;
  }
  const int lds = LDS_BYTES;
  static bool attr_set = false;
  if (!attr_set) {
    hipError_t e = hipFuncSetAttribute((const void*)gemm_rs, hipFuncAttributeMaxDynamicSharedMemorySize, lds);
    if (e != hipSuccess) { fprintf(stderr, "hipFuncSetAttribute: %s\n", hipGetErrorString(e)); return; }
    attr_set = true;
  }
  Args a;
  for (int p = 0; p < 8; ++p) a.ws[p] = (char*)d_ws_all[rank ^ ((p + 2) & 7)];
  a.my = (char*)d_ws_all[rank]; a.x = (const char*)d_in[0]; a.w = (const char*)d_in[1]; a.out = (char*)d_out; a.rank = rank;
  gemm_rs<<<256, 512, lds, stream>>>(a);
}
```

```cpp
#include <hip/hip_runtime.h>
#include <rccl/rccl.h>
#include <cstdio>

typedef unsigned u32x4 __attribute__((ext_vector_type(4)));
typedef unsigned u32x2 __attribute__((ext_vector_type(2)));
typedef short s16x4 __attribute__((ext_vector_type(4)));
typedef __bf16 bf16x8 __attribute__((ext_vector_type(8)));
typedef __bf16 bf16x2 __attribute__((ext_vector_type(2)));
typedef float f32x2 __attribute__((ext_vector_type(2)));
typedef float f32x4 __attribute__((ext_vector_type(4)));
typedef unsigned long long u64;
typedef __amdgpu_buffer_rsrc_t rsrc_t;
#define LDSP __attribute__((address_space(3)))
#define CLOBBER() asm volatile("" ::: "memory")

constexpr unsigned KIB = 1024u, MIB = 1024u * 1024u;
constexpr unsigned SLOT_BYTES = 512 * KIB;
constexpr unsigned IN_BYTES = 6 * SLOT_BYTES;
constexpr unsigned PART_BYTES = 2 * MIB;
constexpr unsigned ROT_BYTES = IN_BYTES + PART_BYTES;
constexpr unsigned CTR_OFF = 3 * ROT_BYTES;
constexpr unsigned ANC_OFF = CTR_OFF + 4096u;
constexpr unsigned WS_NEED = CTR_OFF + 64 * KIB;
constexpr unsigned X_BYTES = 2048u * 256u * 2u, W_BYTES = 256u * 2048u * 2u, Y_BYTES = 256u * 2048u * 2u;
constexpr u64 DEADLINE = 100000000ull;
constexpr unsigned DS_IN = 434u, DS_OUT = 438u;
constexpr unsigned XSLOTS = 224u, WSLOTS = 256u;
constexpr unsigned LEAD = 20u;
constexpr unsigned T_LOCAL = 120u;
constexpr unsigned Z_STEP = 12u;
constexpr int SYS = 17, DEV = 16;
constexpr int LDS_TW = 128 * 1024, LDS_BYTES = LDS_TW + 64;

struct Args {
  const char* x; char* my; const char* w; char* out;
  long long rank;
  char* ws[8];
};

__device__ __forceinline__ u64 now() { return __builtin_amdgcn_s_memrealtime(); }
__device__ __forceinline__ rsrc_t mk(const void* p, unsigned bytes) { return __builtin_amdgcn_make_buffer_rsrc((void*)p, 0, bytes, 0x00020000); }
__device__ __forceinline__ unsigned umin2(unsigned a, unsigned b) { return a < b ? a : b; }
__device__ __forceinline__ unsigned min4(u32x4 v) { return umin2(umin2(v.x, v.y), umin2(v.z, v.w)); }
__device__ __forceinline__ bool all_nz(unsigned m) { return __builtin_amdgcn_ballot_w64(m == 0u) == 0ull; }
__device__ __forceinline__ u32x4 nz(u32x4 v) {
  v.x = v.x ? v.x : 0x8000u; v.y = v.y ? v.y : 0x8000u; v.z = v.z ? v.z : 0x8000u; v.w = v.w ? v.w : 0x8000u; return v;
}
__device__ __forceinline__ unsigned pack2(float a, float b) { f32x2 f = {a, b}; return __builtin_bit_cast(unsigned, __builtin_convertvector(f, bf16x2)); }
__device__ __forceinline__ float lo16(unsigned u) { return __builtin_bit_cast(float, u << 16); }
__device__ __forceinline__ float hi16(unsigned u) { return __builtin_bit_cast(float, u & 0xffff0000u); }
__device__ __forceinline__ float silu(float y) {
  const float c = __builtin_amdgcn_fmed3f(y, -60.f, 60.f);
  const float e = __builtin_amdgcn_exp2f(-1.4426950408889634f * c);
  return y * __builtin_amdgcn_rcpf(1.f + e);
}
template <int AUX> __device__ __forceinline__ void st128(u32x4 v, rsrc_t r, unsigned off) { __builtin_amdgcn_raw_buffer_store_b128(v, r, off, 0, AUX); }
template <int AUX> __device__ __forceinline__ u32x4 ld128(rsrc_t r, unsigned voff, unsigned soff) { return __builtin_amdgcn_raw_buffer_load_b128(r, voff, soff, AUX); }
__device__ __forceinline__ void wait_until(u64 t0, unsigned ticks) {
  for (;;) {
    const unsigned el = (unsigned)(now() - t0);
    if (el >= ticks) break;
    if (ticks - el > 60u) __builtin_amdgcn_s_sleep(16); else __builtin_amdgcn_s_sleep(1);
  }
}
__device__ __forceinline__ f32x4 mfma(u32x4 wf, u32x4 xf, f32x4 c) {
  return __builtin_amdgcn_mfma_f32_16x16x32_bf16(__builtin_bit_cast(bf16x8, wf), __builtin_bit_cast(bf16x8, xf), c, 0, 0, 0);
}
__device__ __forceinline__ unsigned wimg(unsigned row, unsigned c16) { return row * 128u + ((c16 ^ ((row & 2u) | ((row >> 1) & 4u))) << 4); }
__device__ __forceinline__ u32x4 tr_frag(const char* img, unsigned row0, unsigned ni, unsigned lane) {
  const unsigned g = lane >> 4, q = (lane & 15u) >> 2, p = lane & 3u;
  const unsigned r0 = row0 + 8u * g + q;
  const s16x4 a = __builtin_amdgcn_ds_read_tr16_b64_v4i16((LDSP s16x4*)(img + wimg(r0, 2u * ni + (p >> 1)) + 8u * (p & 1u)));
  const s16x4 b = __builtin_amdgcn_ds_read_tr16_b64_v4i16((LDSP s16x4*)(img + wimg(r0 + 4u, 2u * ni + (p >> 1)) + 8u * (p & 1u)));
  const u32x2 ua = __builtin_bit_cast(u32x2, a), ub = __builtin_bit_cast(u32x2, b);
  return (u32x4){ua.x, ua.y, ub.x, ub.y};
}

__global__ void __launch_bounds__(512, 2) gemm_rs(Args a) {
  extern __shared__ __attribute__((aligned(16))) char smem[];
  const unsigned b = blockIdx.x;
  char* p_dst = a.ws[(b >> 3) & 7u]; char* p_my = a.my; const char* p_x = a.x; const char* p_w = a.w; char* p_out = a.out;
  unsigned p_rank = (unsigned)a.rank;
  const u64 t0 = now();
  asm volatile("" : "+s"(p_dst), "+s"(p_my), "+s"(p_x), "+s"(p_w), "+s"(p_out), "+s"(p_rank));
  const unsigned lane = threadIdx.x & 63u;
  const unsigned wave = __builtin_amdgcn_readfirstlane(threadIdx.x >> 6);
  const unsigned c = 3u - ((b >> 6) & 3u), qrel = ((b >> 3) + 2u) & 7u, j = (b + qrel) & 7u;
  const unsigned rt = j >> 2, tj = 4u * c + (j & 3u), w24 = c * 8u + j;
  const bool xsnd = b < 48u, wsnd = qrel == 0u && c < 3u;
  const unsigned erel = wsnd ? 2u + w24 % 6u : qrel;
  const unsigned rank = p_rank & 7u;
  const unsigned rm = rank >> 2, rn = (rank >> 1) & 1u, rk = rank & 1u, q = rank ^ qrel;
  const rsrc_t rs_my = mk(p_my, WS_NEED), rs_d = mk(p_dst, WS_NEED), rs_x = mk(p_x, X_BYTES), rs_w = mk(p_w, W_BYTES);
  const unsigned callv = __builtin_amdgcn_raw_buffer_load_b32(rs_my, CTR_OFF + b * 4u, 0, SYS);
  unsigned call;
  unsigned t_w0 = 80u + ((XSLOTS * DS_IN) >> 8);
  auto rebase = [&]() {
    wait_until(t0, 300u);
    const unsigned an = __builtin_amdgcn_raw_buffer_load_b32(rs_my, ANC_OFF + (erel < 2u ? 2u : erel) * 128u + (lane & 7u) * 4u, 0, SYS);
    int d = (int)(an - (unsigned)t0);
    d = (d < -200 || d > 400) ? 1000 : d;
    int v = d * 256;
    #pragma unroll
    for (int o = 1; o < 8; ++o) { const int e = __shfl_xor(d, o); v -= (e < d || (e == d && ((lane ^ o) & 7u) < (lane & 7u))) ? (int)(4u * DS_IN) : 0; }
    v = d == 1000 ? -(1 << 20) : v;
    #pragma unroll
    for (int o = 1; o < 8; o <<= 1) { const int e = __shfl_xor(v, o); v = e > v ? e : v; }
    v = __builtin_amdgcn_readfirstlane(v);
    if (v > -(1 << 20)) t_w0 = (unsigned)((v + (int)(XSLOTS * DS_IN)) >> 8);
  };

  if (__builtin_expect(xsnd, 1)) {
    u32x4 xs[4];
    if (wave != 0u) __builtin_amdgcn_s_sleep(6);
    #pragma unroll
    for (int ks = 0; ks < 4; ++ks)
      xs[ks] = ld128<0>(rs_x, ((lane & 15u) * 256u + ks * 32u + (lane >> 4) * 8u) * 2u, (((q ^ wave) * 256u + (q >> 2) * 128u + j * 16u) * 256u + (q & 1u) * 128u) * 2u);
    CLOBBER();
    if (wave != 0u) {
      __builtin_amdgcn_s_barrier();
      const unsigned xbase = (unsigned)(now() - t0) - LEAD;
      if (wave == 7u && lane == 0u) __builtin_amdgcn_raw_buffer_store_b32((unsigned)t0 + xbase, rs_my, ANC_OFF + qrel * 128u + j * 4u, 0, SYS);
      t_w0 = xbase + ((XSLOTS * DS_IN) >> 8);
      wait_until(t0, xbase + (((wave - (wave > qrel ? 1u : 0u)) * 32u * DS_IN) >> 8));
    }
    CLOBBER();
    call = __builtin_amdgcn_readfirstlane(callv) + 1u;
    if (wave != qrel) {
      const unsigned dbase = (call % 3u) * ROT_BYTES + (qrel - 2u) * SLOT_BYTES + (wave * 8u + j) * 4096u + lane * 16u;
      #pragma unroll
      for (int ks = 0; ks < 4; ++ks) st128<SYS>(nz(xs[ks]), rs_d, dbase + ks * 1024u);
    }
    if (wave == 0u) {
      CLOBBER();
      __builtin_amdgcn_s_barrier();
      t_w0 = (unsigned)(now() - t0) + ((XSLOTS * DS_IN) >> 8);
    }
    wait_until(t0, t_w0);
  } else if (wsnd) {
    const unsigned d = rank ^ erel, k4 = w24 / 6u;
    const rsrc_t rs_e = mk(a.ws[(erel - 2u) & 7u], WS_NEED);
    wait_until(t0, T_LOCAL);
    u32x4 wl[2][4];
    #pragma unroll
    for (int h = 0; h < 2; ++h)
      #pragma unroll
      for (int i = 0; i < 4; ++i)
        wl[h][i] = ld128<0>(rs_w, ((lane >> 3) * 2048u + (lane & 7u) * 8u + h * 64u) * 2u,
                            (((d & 1u) * 128u + k4 * 32u + i * 8u) * 2048u + ((d >> 1) & 1u) * 1024u + wave * 128u) * 2u);
    call = __builtin_amdgcn_readfirstlane(callv) + 1u;
    const unsigned dbase = (call % 3u) * ROT_BYTES + (erel - 2u) * SLOT_BYTES + 256u * KIB + (wave * 8u + k4) * 4096u + lane * 16u;
    char* img = smem + wave * 16384u;
    #pragma unroll
    for (int h = 0; h < 2; ++h)
      #pragma unroll
      for (int i = 0; i < 4; ++i) *(LDSP u32x4*)(img + h * 4096u + wimg(i * 8u + (lane >> 3), lane & 7u)) = wl[h][i];
    #pragma unroll
    for (int h = 0; h < 2; ++h)
      #pragma unroll
      for (int ni = 0; ni < 4; ++ni) wl[h][ni] = nz(tr_frag(img + h * 4096u, 0, ni, lane));
    rebase();
    #pragma unroll
    for (int h = 0; h < 2; ++h) {
      wait_until(t0, t_w0 + (((wave * 8u + k4 + h * 4u) * 4u * DS_IN) >> 8));
      #pragma unroll
      for (int ni = 0; ni < 4; ++ni) st128<SYS>(wl[h][ni], rs_e, dbase + h * 16384u + ni * 1024u);
    }
    wait_until(t0, t_w0 + ((WSLOTS * DS_IN) >> 8));
  } else {
    call = __builtin_amdgcn_readfirstlane(callv) + 1u;
  }

  const unsigned rot = call % 3u;
  const bool fin = qrel == 0u || (qrel >= 2u && c < 2u);
  const unsigned omh = rm ^ (qrel >> 2), onh = rn ^ ((qrel >> 1) & 1u);
  const unsigned otj = qrel == 0u ? tj : 4u * ((c == 0u ? 2u : 0u) + (qrel & 1u)) + (j & 3u);
  const bool extra = wave + 2u == qrel;
  f32x4 acc[4][4];
  if (wave < 6u && !extra) {
    u32x4 xf[4][4], wf[4][4];
    bool dead = false;
    const unsigned sbase = rot * ROT_BYTES + wave * SLOT_BYTES;
    const unsigned xblk = (qrel * 2u + rt) * 16384u, wblk = 256u * KIB + tj * 16384u;
    const unsigned moff = (lane < 32u ? xblk : wblk) + ((lane >> 3) & 3u) * 4096u + 4064u + (lane & 7u) * 4u;
    auto poll = [&](u64 need) -> u64 {
      for (;;) {
        const u64 miss = __builtin_amdgcn_ballot_w64(__builtin_amdgcn_raw_buffer_load_b32(rs_my, moff, sbase, SYS) == 0u);
        if ((miss & need) == 0ull) return miss;
        if (now() - t0 > DEADLINE) { dead = true; return 0ull; }
        __builtin_amdgcn_s_sleep(3); CLOBBER();
      }
    };
    bool xok = false;
    for (;;) {
      unsigned m = ~0u;
      if (!xok) {
        const u64 miss = poll(0xffffffffull);
        #pragma unroll
        for (int i = 0; i < 16; ++i) xf[i >> 2][i & 3] = ld128<SYS>(rs_my, lane * 16u + (i & 3) * 1024u, sbase + xblk + (i >> 2) * 4096u);
        if (miss != 0ull) {
          #pragma unroll
          for (int i = 0; i < 16; ++i) m = umin2(m, min4(xf[i >> 2][i & 3]));
          if (!all_nz(m) && now() - t0 <= DEADLINE) { __builtin_amdgcn_s_sleep(4); CLOBBER(); continue; }
          poll(0xffffffff00000000ull);
        }
      }
      CLOBBER();
      #pragma unroll
      for (int i = 0; i < 16; ++i) wf[i >> 2][i & 3] = ld128<SYS>(rs_my, lane * 16u + (i & 3) * 1024u, sbase + wblk + (i >> 2) * 4096u);
      unsigned mx = ~0u;
      #pragma unroll
      for (int ks = 0; ks < 4; ++ks)
        #pragma unroll
        for (int ni = 0; ni < 4; ++ni) {
          __builtin_amdgcn_sched_barrier(0);
          #pragma unroll
          for (int mi = 0; mi < 4; ++mi) acc[mi][ni] = mfma(wf[ks][ni], xf[mi][ks], ks ? acc[mi][ni] : (f32x4){0.f, 0.f, 0.f, 0.f});
          m = umin2(m, min4(wf[ks][ni]));
          mx = umin2(mx, min4(xf[ni][ks]));
        }
      xok = all_nz(mx);
      if (xok && all_nz(m)) break;
      if (dead || now() - t0 > DEADLINE) { dead = true; break; }
    }
    if (dead) acc[0][0] = acc[1][0] = acc[2][0] = acc[3][0] = (f32x4){__builtin_nanf(""), __builtin_nanf(""), __builtin_nanf(""), __builtin_nanf("")};
  } else {
    #pragma unroll
    for (int i = 0; i < 16; ++i) acc[i >> 2][i & 3] = (f32x4){0.f, 0.f, 0.f, 0.f};
    wait_until(t0, T_LOCAL);
    const unsigned lrow = extra ? rank * 256u + omh * 128u + rt * 64u : q * 256u + rm * 128u + rt * 64u;
    const unsigned lcol = extra ? onh * 1024u + otj * 64u : rn * 1024u + tj * 64u;
    const unsigned k0 = extra ? 0u : (wave - 6u) * 128u, k1 = extra ? (fin ? 256u : 0u) : k0 + 128u;
    char* img = smem + wave * 16384u;
    const bool zer = !extra && qrel != 0u && (c == 1u || c == 2u);
    #pragma unroll 1
    for (unsigned ph = 0; ph < 2u; ++ph) {
      if ((ph ^ wave) & 1u) {
        if (zer) {
          const unsigned zi = (((qrel - 1u) * 2u + (c - 1u)) * 8u + j) * 2u + (wave - 6u);
          const unsigned zb = ((call + 2u) % 3u) * ROT_BYTES + lane * 16u;
          unsigned tz = (unsigned)(now() - t0) + (zi & 15u);
          #pragma unroll 1
          for (unsigned i = zi; i < ROT_BYTES / 1024u; i += 224u) {
            wait_until(t0, tz); tz += Z_STEP;
            st128<DEV>((u32x4){0u, 0u, 0u, 0u}, rs_my, zb + i * 1024u);
          }
        }
      } else {
        #pragma unroll 1
        for (unsigned k = k0; k < k1; k += 128u) {
          u32x4 wr[16], xf[4][4];
          #pragma unroll
          for (int i = 0; i < 16; ++i) wr[i] = ld128<0>(rs_w, ((lane >> 3) * 2048u + (lane & 7u) * 8u) * 2u, ((k + i * 8u) * 2048u + lcol) * 2u);
          #pragma unroll
          for (int mi = 0; mi < 4; ++mi)
            #pragma unroll
            for (int ks = 0; ks < 4; ++ks)
              xf[mi][ks] = ld128<0>(rs_x, ((lane & 15u) * 256u + ks * 32u + (lane >> 4) * 8u) * 2u, ((lrow + mi * 16u) * 256u + k) * 2u);
          __builtin_amdgcn_sched_barrier(0);
          #pragma unroll
          for (int i = 0; i < 16; ++i) *(LDSP u32x4*)(img + wimg(i * 8u + (lane >> 3), lane & 7u)) = wr[i];
          #pragma unroll
          for (int ks = 0; ks < 4; ++ks)
            #pragma unroll
            for (int ni = 0; ni < 4; ++ni) {
              const u32x4 wf = tr_frag(img, ks * 32u, ni, lane);
              #pragma unroll
              for (int mi = 0; mi < 4; ++mi) acc[mi][ni] = mfma(wf, xf[mi][ks], acc[mi][ni]);
            }
          CLOBBER();
        }
      }
    }
    if (wave == 7u && qrel != 0u) { rebase(); if (lane == 0u) *(LDSP unsigned*)(smem + LDS_TW) = t_w0 + (unsigned)t0; }
  }
  {
    char* slab = smem + wave * 16384u;
    #pragma unroll
    for (int mi = 0; mi < 4; ++mi)
      #pragma unroll
      for (int ni = 0; ni < 4; ++ni) {
        const unsigned nq = ni * 4u + (lane >> 4);
        *(LDSP f32x4*)(slab + (((mi * 16u + nq) * 16u + ((lane & 15u) ^ (nq & 14u))) << 4)) = acc[mi][ni];
      }
  }
  asm volatile("s_waitcnt lgkmcnt(0)" ::: "memory");
  __builtin_amdgcn_s_barrier();
  CLOBBER();
  if (threadIdx.x == 0) __builtin_amdgcn_raw_buffer_store_b32(call, rs_my, CTR_OFF + b * 4u, 0, 0);
  f32x4 s0, s1, e0, e1;
  {
    const unsigned m = wave * 8u + (lane >> 3), o = lane & 7u;
    const unsigned ad = (((m >> 4) * 16u + 2u * o) * 16u + ((m & 15u) ^ (2u * o))) << 4;
    const unsigned sx = (qrel - 2u) & 7u;
    e0 = *(LDSP f32x4*)(smem + sx * 16384u + ad);
    e1 = *(LDSP f32x4*)(smem + sx * 16384u + ad + 256u);
    s0 = *(LDSP f32x4*)(smem + (sx ^ 1u) * 16384u + ad);
    s1 = *(LDSP f32x4*)(smem + (sx ^ 1u) * 16384u + ad + 256u);
    #pragma unroll
    for (int s = 2; s < 8; ++s) {
      s0 += *(LDSP f32x4*)(smem + (sx ^ s) * 16384u + ad);
      s1 += *(LDSP f32x4*)(smem + (sx ^ s) * 16384u + ad + 256u);
    }
    if (qrel < 2u) { s0 += e0; s1 += e1; e0 = s0; e1 = s1; }
  }
  const unsigned strip = wave * 1024u + lane * 16u;
  if (qrel != 0u) {
    const u32x4 p = {pack2(s0.x, s0.y), pack2(s0.z, s0.w), pack2(s1.x, s1.y), pack2(s1.z, s1.w)};
    const unsigned tb = __builtin_amdgcn_readfirstlane(*(LDSP unsigned*)(smem + LDS_TW)) - (unsigned)t0 + (qrel == 1u ? 0u : (WSLOTS * DS_IN) >> 8);
    wait_until(t0, tb + ((((tj * 2u + rt) * 8u + wave) * DS_OUT) >> 8));
    st128<SYS>(nz(p), rs_d, strip + rot * ROT_BYTES + IN_BYTES + rk * MIB + (((rm * 2u + rt) * 32u + rn * 16u + tj) << 13));
  }
  if (fin) {
    const unsigned tbase = rot * ROT_BYTES + IN_BYTES + (((omh * 2u + rt) * 32u + onh * 16u + otj) << 13);
    const unsigned pa = tbase + (rk ^ 1u) * MIB, pb = tbase + (qrel != 0u ? rk : (rk ^ 1u)) * MIB;
    u32x4 u, v;
    bool dead = false;
    for (unsigned it = 1;; ++it) {
      u = ld128<SYS>(rs_my, strip, pa);
      v = ld128<SYS>(rs_my, strip, pb);
      if (all_nz(umin2(min4(u), min4(v)))) break;
      if ((it & 63u) == 0u && now() - t0 > DEADLINE) { dead = true; break; }
      __builtin_amdgcn_s_sleep(1);
      CLOBBER();
    }
    if (qrel == 0u) v = (u32x4){0u, 0u, 0u, 0u};
    const float y0 = e0.x + lo16(u.x) + lo16(v.x), y1 = e0.y + hi16(u.x) + hi16(v.x), y2 = e0.z + lo16(u.y) + lo16(v.y), y3 = e0.w + hi16(u.y) + hi16(v.y);
    const float y4 = e1.x + lo16(u.z) + lo16(v.z), y5 = e1.y + hi16(u.z) + hi16(v.z), y6 = e1.z + lo16(u.w) + lo16(v.w), y7 = e1.w + hi16(u.w) + hi16(v.w);
    u32x4 o = {pack2(silu(y0), silu(y1)), pack2(silu(y2), silu(y3)), pack2(silu(y4), silu(y5)), pack2(silu(y6), silu(y7))};
    if (dead) o = (u32x4){0x7fc07fc0u, 0x7fc07fc0u, 0x7fc07fc0u, 0x7fc07fc0u};
    const unsigned ooff = ((omh * 128u + rt * 64u + wave * 8u + (lane >> 3)) * 2048u + onh * 1024u + otj * 64u + (lane & 7u) * 8u) * 2u;
    if (otj < 12u && qrel != 0u) st128<DEV>(o, mk(p_out, Y_BYTES), ooff);
    else st128<0>(o, mk(p_out, Y_BYTES), ooff);
  }
}

extern "C" void kernel_launch(int rank, int nranks, ncclComm_t comm, hipStream_t stream,
                              void* const* d_in, const int* in_sizes, int n_in,
                              void* d_out, int out_size, const int* in_dtypes, int out_dtype,
                              void* const* d_out_all, void* const* d_ws_all, size_t ws_bytes) {
  (void)comm; (void)in_dtypes; (void)out_dtype; (void)d_out_all;
  if (nranks != 8 || (unsigned)rank >= 8u || n_in < 2 || ws_bytes < WS_NEED || (size_t)in_sizes[0] * 2 != X_BYTES ||
      (size_t)in_sizes[1] * 2 != W_BYTES || (size_t)out_size * 2 != Y_BYTES) {
    fprintf(stderr, "kernel_launch: unexpected configuration\n");
    return;
  }
  const int lds = LDS_BYTES;
  static bool attr_set = false;
  if (!attr_set) {
    hipError_t e = hipFuncSetAttribute((const void*)gemm_rs, hipFuncAttributeMaxDynamicSharedMemorySize, lds);
    if (e != hipSuccess) { fprintf(stderr, "hipFuncSetAttribute: %s\n", hipGetErrorString(e)); return; }
    attr_set = true;
  }
  Args a;
  for (int p = 0; p < 8; ++p) a.ws[p] = (char*)d_ws_all[rank ^ ((p + 2) & 7)];
  a.my = (char*)d_ws_all[rank]; a.x = (const char*)d_in[0]; a.w = (const char*)d_in[1]; a.out = (char*)d_out; a.rank = rank;
  gemm_rs<<<256, 512, lds, stream>>>(a);
}
```

```cpp
#include <hip/hip_runtime.h>
#include <rccl/rccl.h>
#include <cstdio>

typedef unsigned u32x4 __attribute__((ext_vector_type(4)));
typedef unsigned u32x2 __attribute__((ext_vector_type(2)));
typedef short s16x4 __attribute__((ext_vector_type(4)));
typedef __bf16 bf16x8 __attribute__((ext_vector_type(8)));
typedef __bf16 bf16x2 __attribute__((ext_vector_type(2)));
typedef float f32x2 __attribute__((ext_vector_type(2)));
typedef float f32x4 __attribute__((ext_vector_type(4)));
typedef unsigned long long u64;
typedef __amdgpu_buffer_rsrc_t rsrc_t;
#define LDSP __attribute__((address_space(3)))
#define CLOBBER() asm volatile("" ::: "memory")

constexpr unsigned KIB = 1024u, MIB = 1024u * 1024u;
constexpr unsigned SLOT_BYTES = 512 * KIB;
constexpr unsigned IN_BYTES = 6 * SLOT_BYTES;
constexpr unsigned PART_BYTES = 2 * MIB;
constexpr unsigned ROT_BYTES = IN_BYTES + PART_BYTES;
constexpr unsigned CTR_OFF = 3 * ROT_BYTES;
constexpr unsigned ANC_OFF = CTR_OFF + 4096u;
constexpr unsigned WS_NEED = CTR_OFF + 64 * KIB;
constexpr unsigned X_BYTES = 2048u * 256u * 2u, W_BYTES = 256u * 2048u * 2u, Y_BYTES = 256u * 2048u * 2u;
constexpr u64 DEADLINE = 100000000ull;
constexpr unsigned DS_IN = 434u, DS_OUT = 438u;
constexpr unsigned XSLOTS = 224u, WSLOTS = 256u;
constexpr unsigned LEAD = 20u;
constexpr unsigned T_LOCAL = 120u;
constexpr unsigned Z_STEP = 12u;
#define PFB 6u
constexpr unsigned CODE_PF = 11648u;
constexpr int SYS = 17, DEV = 16;
constexpr int LDS_TW = 128 * 1024, LDS_BYTES = LDS_TW + 64;

struct Args {
  const char* x; char* my; const char* w; char* out;
  long long rank;
  char* ws[8];
};

__device__ __forceinline__ u64 now() { return __builtin_amdgcn_s_memrealtime(); }
__device__ __forceinline__ rsrc_t mk(const void* p, unsigned bytes) { return __builtin_amdgcn_make_buffer_rsrc((void*)p, 0, bytes, 0x00020000); }
__device__ __forceinline__ unsigned umin2(unsigned a, unsigned b) { return a < b ? a : b; }
__device__ __forceinline__ unsigned min4(u32x4 v) { return umin2(umin2(v.x, v.y), umin2(v.z, v.w)); }
__device__ __forceinline__ bool all_nz(unsigned m) { return __builtin_amdgcn_ballot_w64(m == 0u) == 0ull; }
__device__ __forceinline__ u32x4 nz(u32x4 v) {
  v.x = v.x ? v.x : 0x8000u; v.y = v.y ? v.y : 0x8000u; v.z = v.z ? v.z : 0x8000u; v.w = v.w ? v.w : 0x8000u; return v;
}
__device__ __forceinline__ unsigned pack2(float a, float b) { f32x2 f = {a, b}; return __builtin_bit_cast(unsigned, __builtin_convertvector(f, bf16x2)); }
__device__ __forceinline__ float lo16(unsigned u) { return __builtin_bit_cast(float, u << 16); }
__device__ __forceinline__ float hi16(unsigned u) { return __builtin_bit_cast(float, u & 0xffff0000u); }
__device__ __forceinline__ float silu(float y) {
  const float c = __builtin_amdgcn_fmed3f(y, -60.f, 60.f);
  const float e = __builtin_amdgcn_exp2f(-1.4426950408889634f * c);
  return y * __builtin_amdgcn_rcpf(1.f + e);
}
template <int AUX> __device__ __forceinline__ void st128(u32x4 v, rsrc_t r, unsigned off) { __builtin_amdgcn_raw_buffer_store_b128(v, r, off, 0, AUX); }
template <int AUX> __device__ __forceinline__ u32x4 ld128(rsrc_t r, unsigned voff, unsigned soff) { return __builtin_amdgcn_raw_buffer_load_b128(r, voff, soff, AUX); }
__device__ __forceinline__ void wait_until(u64 t0, unsigned ticks) {
  for (;;) {
    const unsigned el = (unsigned)(now() - t0);
    if (el >= ticks) break;
    if (ticks - el > 60u) __builtin_amdgcn_s_sleep(16); else __builtin_amdgcn_s_sleep(1);
  }
}
__device__ __forceinline__ f32x4 mfma(u32x4 wf, u32x4 xf, f32x4 c) {
  return __builtin_amdgcn_mfma_f32_16x16x32_bf16(__builtin_bit_cast(bf16x8, wf), __builtin_bit_cast(bf16x8, xf), c, 0, 0, 0);
}
__device__ __forceinline__ unsigned wimg(unsigned row, unsigned c16) { return row * 128u + ((c16 ^ ((row & 2u) | ((row >> 1) & 4u))) << 4); }
__device__ __forceinline__ u32x4 tr_frag(const char* img, unsigned row0, unsigned ni, unsigned lane) {
  const unsigned g = lane >> 4, q = (lane & 15u) >> 2, p = lane & 3u;
  const unsigned r0 = row0 + 8u * g + q;
  const s16x4 a = __builtin_amdgcn_ds_read_tr16_b64_v4i16((LDSP s16x4*)(img + wimg(r0, 2u * ni + (p >> 1)) + 8u * (p & 1u)));
  const s16x4 b = __builtin_amdgcn_ds_read_tr16_b64_v4i16((LDSP s16x4*)(img + wimg(r0 + 4u, 2u * ni + (p >> 1)) + 8u * (p & 1u)));
  const u32x2 ua = __builtin_bit_cast(u32x2, a), ub = __builtin_bit_cast(u32x2, b);
  return (u32x4){ua.x, ua.y, ub.x, ub.y};
}

__global__ void __launch_bounds__(512, 2) gemm_rs(Args a) {
  extern __shared__ __attribute__((aligned(16))) char smem[];
  const unsigned b = blockIdx.x;
  char* p_dst = a.ws[(b >> 3) & 7u]; char* p_my = a.my; const char* p_x = a.x; const char* p_w = a.w; char* p_out = a.out;
  unsigned p_rank = (unsigned)a.rank;
  const u64 t0 = now();
  const unsigned pf = __builtin_amdgcn_raw_buffer_load_b32(mk((const void*)((u64)__builtin_amdgcn_s_getpc() & ~255ull), (b >> 3) == PFB ? CODE_PF : 0u),
                                                           threadIdx.x * 32u, 0, 0);
  asm volatile("" : "+s"(p_dst), "+s"(p_my), "+s"(p_x), "+s"(p_w), "+s"(p_out), "+s"(p_rank));
  const unsigned lane = threadIdx.x & 63u;
  const unsigned wave = __builtin_amdgcn_readfirstlane(threadIdx.x >> 6);
  const unsigned c = 3u - ((b >> 6) & 3u), qrel = ((b >> 3) + 2u) & 7u, j = (b + qrel) & 7u;
  const unsigned rt = j >> 2, tj = 4u * c + (j & 3u), w24 = c * 8u + j;
  const bool xsnd = b < 48u, wsnd = qrel == 0u && c < 3u;
  const unsigned erel = wsnd ? 2u + w24 % 6u : qrel;
  const unsigned rank = p_rank & 7u;
  const unsigned rm = rank >> 2, rn = (rank >> 1) & 1u, rk = rank & 1u, q = rank ^ qrel;
  const rsrc_t rs_my = mk(p_my, WS_NEED), rs_d = mk(p_dst, WS_NEED), rs_x = mk(p_x, X_BYTES), rs_w = mk(p_w, W_BYTES);
  const unsigned callv = __builtin_amdgcn_raw_buffer_load_b32(rs_my, CTR_OFF + b * 4u, 0, SYS);
  unsigned call;
  unsigned t_w0 = 80u + ((XSLOTS * DS_IN) >> 8);
  auto rebase = [&]() {
    wait_until(t0, 300u);
    const unsigned an = __builtin_amdgcn_raw_buffer_load_b32(rs_my, ANC_OFF + (erel < 2u ? 2u : erel) * 128u + (lane & 7u) * 4u, 0, SYS);
    int d = (int)(an - (unsigned)t0);
    d = (d < -200 || d > 400) ? 1000 : d;
    int v = d * 256;
    #pragma unroll
    for (int o = 1; o < 8; ++o) { const int e = __shfl_xor(d, o); v -= (e < d || (e == d && ((lane ^ o) & 7u) < (lane & 7u))) ? (int)(4u * DS_IN) : 0; }
    v = d == 1000 ? -(1 << 20) : v;
    #pragma unroll
    for (int o = 1; o < 8; o <<= 1) { const int e = __shfl_xor(v, o); v = e > v ? e : v; }
    v = __builtin_amdgcn_readfirstlane(v);
    if (v > -(1 << 20)) t_w0 = (unsigned)((v + (int)(XSLOTS * DS_IN)) >> 8);
  };

  if (__builtin_expect(xsnd, 1)) {
    u32x4 xs[4];
    if (wave != 0u) __builtin_amdgcn_s_sleep(6);
    #pragma unroll
    for (int ks = 0; ks < 4; ++ks)
      xs[ks] = ld128<0>(rs_x, ((lane & 15u) * 256u + ks * 32u + (lane >> 4) * 8u) * 2u, (((q ^ wave) * 256u + (q >> 2) * 128u + j * 16u) * 256u + (q & 1u) * 128u) * 2u);
    CLOBBER();
    if (wave != 0u) {
      __builtin_amdgcn_s_barrier();
      const unsigned xbase = (unsigned)(now() - t0) - LEAD;
      if (wave == 7u && lane == 0u) __builtin_amdgcn_raw_buffer_store_b32((unsigned)t0 + xbase, rs_my, ANC_OFF + qrel * 128u + j * 4u, 0, SYS);
      t_w0 = xbase + ((XSLOTS * DS_IN) >> 8);
      wait_until(t0, xbase + (((wave - (wave > qrel ? 1u : 0u)) * 32u * DS_IN) >> 8));
    }
    CLOBBER();
    call = __builtin_amdgcn_readfirstlane(callv) + 1u;
    if (wave != qrel) {
      const unsigned dbase = (call % 3u) * ROT_BYTES + (qrel - 2u) * SLOT_BYTES + (wave * 8u + j) * 4096u + lane * 16u;
      #pragma unroll
      for (int ks = 0; ks < 4; ++ks) st128<SYS>(nz(xs[ks]), rs_d, dbase + ks * 1024u);
    }
    if (wave == 0u) {
      CLOBBER();
      __builtin_amdgcn_s_barrier();
      t_w0 = (unsigned)(now() - t0) + ((XSLOTS * DS_IN) >> 8);
    }
    wait_until(t0, t_w0);
  } else if (wsnd) {
    const unsigned d = rank ^ erel, k4 = w24 / 6u;
    const rsrc_t rs_e = mk(a.ws[(erel - 2u) & 7u], WS_NEED);
    wait_until(t0, T_LOCAL);
    u32x4 wl[2][4];
    #pragma unroll
    for (int h = 0; h < 2; ++h)
      #pragma unroll
      for (int i = 0; i < 4; ++i)
        wl[h][i] = ld128<0>(rs_w, ((lane >> 3) * 2048u + (lane & 7u) * 8u + h * 64u) * 2u,
                            (((d & 1u) * 128u + k4 * 32u + i * 8u) * 2048u + ((d >> 1) & 1u) * 1024u + wave * 128u) * 2u);
    call = __builtin_amdgcn_readfirstlane(callv) + 1u;
    const unsigned dbase = (call % 3u) * ROT_BYTES + (erel - 2u) * SLOT_BYTES + 256u * KIB + (wave * 8u + k4) * 4096u + lane * 16u;
    char* img = smem + wave * 16384u;
    #pragma unroll
    for (int h = 0; h < 2; ++h)
      #pragma unroll
      for (int i = 0; i < 4; ++i) *(LDSP u32x4*)(img + h * 4096u + wimg(i * 8u + (lane >> 3), lane & 7u)) = wl[h][i];
    #pragma unroll
    for (int h = 0; h < 2; ++h)
      #pragma unroll
      for (int ni = 0; ni < 4; ++ni) wl[h][ni] = nz(tr_frag(img + h * 4096u, 0, ni, lane));
    rebase();
    #pragma unroll
    for (int h = 0; h < 2; ++h) {
      wait_until(t0, t_w0 + (((wave * 8u + k4 + h * 4u) * 4u * DS_IN) >> 8));
      #pragma unroll
      for (int ni = 0; ni < 4; ++ni) st128<SYS>(wl[h][ni], rs_e, dbase + h * 16384u + ni * 1024u);
    }
    wait_until(t0, t_w0 + ((WSLOTS * DS_IN) >> 8));
  } else {
    call = __builtin_amdgcn_readfirstlane(callv) + 1u;
  }

  asm volatile("" :: "v"(pf));
  const unsigned rot = call % 3u;
  const bool fin = qrel == 0u || (qrel >= 2u && c < 2u);
  const unsigned omh = rm ^ (qrel >> 2), onh = rn ^ ((qrel >> 1) & 1u);
  const unsigned otj = qrel == 0u ? tj : 4u * ((c == 0u ? 2u : 0u) + (qrel & 1u)) + (j & 3u);
  const bool extra = wave + 2u == qrel;
  f32x4 acc[4][4];
  if (wave < 6u && !extra) {
    u32x4 xf[4][4], wf[4][4];
    bool dead = false;
    const unsigned sbase = rot * ROT_BYTES + wave * SLOT_BYTES;
    const unsigned xblk = (qrel * 2u + rt) * 16384u, wblk = 256u * KIB + tj * 16384u;
    const unsigned moff = (lane < 32u ? xblk : wblk) + ((lane >> 3) & 3u) * 4096u + 4064u + (lane & 7u) * 4u;
    auto poll = [&](u64 need) -> u64 {
      for (;;) {
        const u64 miss = __builtin_amdgcn_ballot_w64(__builtin_amdgcn_raw_buffer_load_b32(rs_my, moff, sbase, SYS) == 0u);
        if ((miss & need) == 0ull) return miss;
        if (now() - t0 > DEADLINE) { dead = true; return 0ull; }
        __builtin_amdgcn_s_sleep(3); CLOBBER();
      }
    };
    bool xok = false;
    for (;;) {
      unsigned m = ~0u;
      if (!xok) {
        const u64 miss = poll(0xffffffffull);
        #pragma unroll
        for (int i = 0; i < 16; ++i) xf[i >> 2][i & 3] = ld128<SYS>(rs_my, lane * 16u + (i & 3) * 1024u, sbase + xblk + (i >> 2) * 4096u);
        if (miss != 0ull) {
          #pragma unroll
          for (int i = 0; i < 16; ++i) m = umin2(m, min4(xf[i >> 2][i & 3]));
          if (!all_nz(m) && now() - t0 <= DEADLINE) { __builtin_amdgcn_s_sleep(4); CLOBBER(); continue; }
          poll(0xffffffff00000000ull);
        }
      }
      CLOBBER();
      #pragma unroll
      for (int i = 0; i < 16; ++i) wf[i >> 2][i & 3] = ld128<SYS>(rs_my, lane * 16u + (i & 3) * 1024u, sbase + wblk + (i >> 2) * 4096u);
      unsigned mx = ~0u;
      #pragma unroll
      for (int ks = 0; ks < 4; ++ks)
        #pragma unroll
        for (int ni = 0; ni < 4; ++ni) {
          __builtin_amdgcn_sched_barrier(0);
          #pragma unroll
          for (int mi = 0; mi < 4; ++mi) acc[mi][ni] = mfma(wf[ks][ni], xf[mi][ks], ks ? acc[mi][ni] : (f32x4){0.f, 0.f, 0.f, 0.f});
          m = umin2(m, min4(wf[ks][ni]));
          mx = umin2(mx, min4(xf[ni][ks]));
        }
      xok = all_nz(mx);
      if (xok && all_nz(m)) break;
      if (dead || now() - t0 > DEADLINE) { dead = true; break; }
    }
    if (dead) acc[0][0] = acc[1][0] = acc[2][0] = acc[3][0] = (f32x4){__builtin_nanf(""), __builtin_nanf(""), __builtin_nanf(""), __builtin_nanf("")};
  } else {
    #pragma unroll
    for (int i = 0; i < 16; ++i) acc[i >> 2][i & 3] = (f32x4){0.f, 0.f, 0.f, 0.f};
    wait_until(t0, T_LOCAL);
    const unsigned lrow = extra ? rank * 256u + omh * 128u + rt * 64u : q * 256u + rm * 128u + rt * 64u;
    const unsigned lcol = extra ? onh * 1024u + otj * 64u : rn * 1024u + tj * 64u;
    const unsigned k0 = extra ? 0u : (wave - 6u) * 128u, k1 = extra ? (fin ? 256u : 0u) : k0 + 128u;
    char* img = smem + wave * 16384u;
    const bool zer = !extra && qrel != 0u && (c == 1u || c == 2u);
    #pragma unroll 1
    for (unsigned ph = 0; ph < 2u; ++ph) {
      if ((ph ^ wave) & 1u) {
        if (zer) {
          const unsigned zi = (((qrel - 1u) * 2u + (c - 1u)) * 8u + j) * 2u + (wave - 6u);
          const unsigned zb = ((call + 2u) % 3u) * ROT_BYTES + lane * 16u;
          unsigned tz = (unsigned)(now() - t0) + (zi & 15u);
          #pragma unroll 1
          for (unsigned i = zi; i < ROT_BYTES / 1024u; i += 224u) {
            wait_until(t0, tz); tz += Z_STEP;
            st128<DEV>((u32x4){0u, 0u, 0u, 0u}, rs_my, zb + i * 1024u);
          }
        }
      } else {
        #pragma unroll 1
        for (unsigned k = k0; k < k1; k += 128u) {
          u32x4 wr[16], xf[4][4];
          #pragma unroll
          for (int i = 0; i < 16; ++i) wr[i] = ld128<0>(rs_w, ((lane >> 3) * 2048u + (lane & 7u) * 8u) * 2u, ((k + i * 8u) * 2048u + lcol) * 2u);
          #pragma unroll
          for (int mi = 0; mi < 4; ++mi)
            #pragma unroll
            for (int ks = 0; ks < 4; ++ks)
              xf[mi][ks] = ld128<0>(rs_x, ((lane & 15u) * 256u + ks * 32u + (lane >> 4) * 8u) * 2u, ((lrow + mi * 16u) * 256u + k) * 2u);
          __builtin_amdgcn_sched_barrier(0);
          #pragma unroll
          for (int i = 0; i < 16; ++i) *(LDSP u32x4*)(img + wimg(i * 8u + (lane >> 3), lane & 7u)) = wr[i];
          #pragma unroll
          for (int ks = 0; ks < 4; ++ks)
            #pragma unroll
            for (int ni = 0; ni < 4; ++ni) {
              const u32x4 wf = tr_frag(img, ks * 32u, ni, lane);
              #pragma unroll
              for (int mi = 0; mi < 4; ++mi) acc[mi][ni] = mfma(wf, xf[mi][ks], acc[mi][ni]);
            }
          CLOBBER();
        }
      }
    }
    if (wave == 7u && qrel != 0u) { rebase(); if (lane == 0u) *(LDSP unsigned*)(smem + LDS_TW) = t_w0 + (unsigned)t0; }
  }
  {
    char* slab = smem + wave * 16384u;
    #pragma unroll
    for (int mi = 0; mi < 4; ++mi)
      #pragma unroll
      for (int ni = 0; ni < 4; ++ni) {
        const unsigned nq = ni * 4u + (lane >> 4);
        *(LDSP f32x4*)(slab + (((mi * 16u + nq) * 16u + ((lane & 15u) ^ (nq & 14u))) << 4)) = acc[mi][ni];
      }
  }
  asm volatile("s_waitcnt lgkmcnt(0)" ::: "memory");
  __builtin_amdgcn_s_barrier();
  CLOBBER();
  if (threadIdx.x == 0) __builtin_amdgcn_raw_buffer_store_b32(call, rs_my, CTR_OFF + b * 4u, 0, 0);
  f32x4 s0, s1, e0, e1;
  {
    const unsigned m = wave * 8u + (lane >> 3), o = lane & 7u;
    const unsigned ad = (((m >> 4) * 16u + 2u * o) * 16u + ((m & 15u) ^ (2u * o))) << 4;
    const unsigned sx = (qrel - 2u) & 7u;
    e0 = *(LDSP f32x4*)(smem + sx * 16384u + ad);
    e1 = *(LDSP f32x4*)(smem + sx * 16384u + ad + 256u);
    s0 = *(LDSP f32x4*)(smem + (sx ^ 1u) * 16384u + ad);
    s1 = *(LDSP f32x4*)(smem + (sx ^ 1u) * 16384u + ad + 256u);
    #pragma unroll
    for (int s = 2; s < 8; ++s) {
      s0 += *(LDSP f32x4*)(smem + (sx ^ s) * 16384u + ad);
      s1 += *(LDSP f32x4*)(smem + (sx ^ s) * 16384u + ad + 256u);
    }
    if (qrel < 2u) { s0 += e0; s1 += e1; e0 = s0; e1 = s1; }
  }
  const unsigned strip = wave * 1024u + lane * 16u;
  if (qrel != 0u) {
    const u32x4 p = {pack2(s0.x, s0.y), pack2(s0.z, s0.w), pack2(s1.x, s1.y), pack2(s1.z, s1.w)};
    const unsigned tb = __builtin_amdgcn_readfirstlane(*(LDSP unsigned*)(smem + LDS_TW)) - (unsigned)t0 + (qrel == 1u ? 0u : (WSLOTS * DS_IN) >> 8);
    wait_until(t0, tb + ((((tj * 2u + rt) * 8u + wave) * DS_OUT) >> 8));
    st128<SYS>(nz(p), rs_d, strip + rot * ROT_BYTES + IN_BYTES + rk * MIB + (((rm * 2u + rt) * 32u + rn * 16u + tj) << 13));
  }
  if (fin) {
    const unsigned tbase = rot * ROT_BYTES + IN_BYTES + (((omh * 2u + rt) * 32u + onh * 16u + otj) << 13);
    const unsigned pa = tbase + (rk ^ 1u) * MIB, pb = tbase + (qrel != 0u ? rk : (rk ^ 1u)) * MIB;
    u32x4 u, v;
    bool dead = false;
    for (unsigned it = 1;; ++it) {
      u = ld128<SYS>(rs_my, strip, pa);
      v = ld128<SYS>(rs_my, strip, pb);
      if (all_nz(umin2(min4(u), min4(v)))) break;
      if ((it & 63u) == 0u && now() - t0 > DEADLINE) { dead = true; break; }
      __builtin_amdgcn_s_sleep(1);
      CLOBBER();
    }
    if (qrel == 0u) v = (u32x4){0u, 0u, 0u, 0u};
    const float y0 = e0.x + lo16(u.x) + lo16(v.x), y1 = e0.y + hi16(u.x) + hi16(v.x), y2 = e0.z + lo16(u.y) + lo16(v.y), y3 = e0.w + hi16(u.y) + hi16(v.y);
    const float y4 = e1.x + lo16(u.z) + lo16(v.z), y5 = e1.y + hi16(u.z) + hi16(v.z), y6 = e1.z + lo16(u.w) + lo16(v.w), y7 = e1.w + hi16(u.w) + hi16(v.w);
    u32x4 o = {pack2(silu(y0), silu(y1)), pack2(silu(y2), silu(y3)), pack2(silu(y4), silu(y5)), pack2(silu(y6), silu(y7))};
    if (dead) o = (u32x4){0x7fc07fc0u, 0x7fc07fc0u, 0x7fc07fc0u, 0x7fc07fc0u};
    const unsigned ooff = ((omh * 128u + rt * 64u + wave * 8u + (lane >> 3)) * 2048u + onh * 1024u + otj * 64u + (lane & 7u) * 8u) * 2u;
    if (otj < 12u && qrel != 0u) st128<DEV>(o, mk(p_out, Y_BYTES), ooff);
    else st128<0>(o, mk(p_out, Y_BYTES), ooff);
  }
}

extern "C" void kernel_launch(int rank, int nranks, ncclComm_t comm, hipStream_t stream,
                              void* const* d_in, const int* in_sizes, int n_in,
                              void* d_out, int out_size, const int* in_dtypes, int out_dtype,
                              void* const* d_out_all, void* const* d_ws_all, size_t ws_bytes) {
  (void)comm; (void)in_dtypes; (void)out_dtype; (void)d_out_all;
  if (nranks != 8 || (unsigned)rank >= 8u || n_in < 2 || ws_bytes < WS_NEED || (size_t)in_sizes[0] * 2 != X_BYTES ||
      (size_t)in_sizes[1] * 2 != W_BYTES || (size_t)out_size * 2 != Y_BYTES) {
    fprintf(stderr, "kernel_launch: unexpected configuration\n");
    return;
  }
  const int lds = LDS_BYTES;
  static bool attr_set = false;
  if (!attr_set) {
    hipError_t e = hipFuncSetAttribute((const void*)gemm_rs, hipFuncAttributeMaxDynamicSharedMemorySize, lds);
    if (e != hipSuccess) { fprintf(stderr, "hipFuncSetAttribute: %s\n", hipGetErrorString(e)); return; }
    attr_set = true;
  }
  Args a;
  for (int p = 0; p < 8; ++p) a.ws[p] = (char*)d_ws_all[rank ^ ((p + 2) & 7)];
  a.my = (char*)d_ws_all[rank]; a.x = (const char*)d_in[0]; a.w = (const char*)d_in[1]; a.out = (char*)d_out; a.rank = rank;
  gemm_rs<<<256, 512, lds, stream>>>(a);
}
```

```cpp
#include <hip/hip_runtime.h>
#include <rccl/rccl.h>
#include <cstdio>

typedef unsigned u32x4 __attribute__((ext_vector_type(4)));
typedef unsigned u32x2 __attribute__((ext_vector_type(2)));
typedef short s16x4 __attribute__((ext_vector_type(4)));
typedef __bf16 bf16x8 __attribute__((ext_vector_type(8)));
typedef __bf16 bf16x2 __attribute__((ext_vector_type(2)));
typedef float f32x2 __attribute__((ext_vector_type(2)));
typedef float f32x4 __attribute__((ext_vector_type(4)));
typedef unsigned long long u64;
typedef __amdgpu_buffer_rsrc_t rsrc_t;
#define LDSP __attribute__((address_space(3)))
#define CLOBBER() asm volatile("" ::: "memory")

constexpr unsigned KIB = 1024u, MIB = 1024u * 1024u;
constexpr unsigned SLOT_BYTES = 512 * KIB;
constexpr unsigned IN_BYTES = 6 * SLOT_BYTES;
constexpr unsigned PART_BYTES = 2 * MIB;
constexpr unsigned ROT_BYTES = IN_BYTES + PART_BYTES;
constexpr unsigned CTR_OFF = 3 * ROT_BYTES;
constexpr unsigned ANC_OFF = CTR_OFF + 4096u;
constexpr unsigned WS_NEED = CTR_OFF + 64 * KIB;
constexpr unsigned X_BYTES = 2048u * 256u * 2u, W_BYTES = 256u * 2048u * 2u, Y_BYTES = 256u * 2048u * 2u;
constexpr u64 DEADLINE = 100000000ull;
constexpr unsigned DS_IN = 434u, DS_OUT = 438u;
constexpr unsigned XSLOTS = 224u, WSLOTS = 256u;
constexpr unsigned LEAD = 20u;
constexpr unsigned T_LOCAL = 120u;
constexpr unsigned Z_STEP = 12u;
constexpr unsigned CODE_PF = 11904u;
constexpr int SYS = 17, DEV = 16;
constexpr int LDS_TW = 128 * 1024, LDS_BYTES = LDS_TW + 64;

struct Args {
  const char* x; char* my; const char* w; char* out;
  long long rank;
  char* ws[8];
};

__device__ __forceinline__ u64 now() { return __builtin_amdgcn_s_memrealtime(); }
__device__ __forceinline__ rsrc_t mk(const void* p, unsigned bytes) { return __builtin_amdgcn_make_buffer_rsrc((void*)p, 0, bytes, 0x00020000); }
__device__ __forceinline__ unsigned umin2(unsigned a, unsigned b) { return a < b ? a : b; }
__device__ __forceinline__ unsigned min4(u32x4 v) { return umin2(umin2(v.x, v.y), umin2(v.z, v.w)); }
__device__ __forceinline__ bool all_nz(unsigned m) { return __builtin_amdgcn_ballot_w64(m == 0u) == 0ull; }
__device__ __forceinline__ u32x4 nz(u32x4 v) {
  v.x = v.x ? v.x : 0x8000u; v.y = v.y ? v.y : 0x8000u; v.z = v.z ? v.z : 0x8000u; v.w = v.w ? v.w : 0x8000u; return v;
}
__device__ __forceinline__ unsigned pack2(float a, float b) { f32x2 f = {a, b}; return __builtin_bit_cast(unsigned, __builtin_convertvector(f, bf16x2)); }
__device__ __forceinline__ float lo16(unsigned u) { return __builtin_bit_cast(float, u << 16); }
__device__ __forceinline__ float hi16(unsigned u) { return __builtin_bit_cast(float, u & 0xffff0000u); }
__device__ __forceinline__ float silu(float y) {
  const float c = __builtin_amdgcn_fmed3f(y, -60.f, 60.f);
  const float e = __builtin_amdgcn_exp2f(-1.4426950408889634f * c);
  return y * __builtin_amdgcn_rcpf(1.f + e);
}
template <int AUX> __device__ __forceinline__ void st128(u32x4 v, rsrc_t r, unsigned off) { __builtin_amdgcn_raw_buffer_store_b128(v, r, off, 0, AUX); }
template <int AUX> __device__ __forceinline__ u32x4 ld128(rsrc_t r, unsigned voff, unsigned soff) { return __builtin_amdgcn_raw_buffer_load_b128(r, voff, soff, AUX); }
__device__ __forceinline__ void wait_until(u64 t0, unsigned ticks) {
  for (;;) {
    const unsigned el = (unsigned)(now() - t0);
    if (el >= ticks) break;
    if (ticks - el > 60u) __builtin_amdgcn_s_sleep(16); else __builtin_amdgcn_s_sleep(1);
  }
}
__device__ __forceinline__ f32x4 mfma(u32x4 wf, u32x4 xf, f32x4 c) {
  return __builtin_amdgcn_mfma_f32_16x16x32_bf16(__builtin_bit_cast(bf16x8, wf), __builtin_bit_cast(bf16x8, xf), c, 0, 0, 0);
}
__device__ __forceinline__ unsigned wimg(unsigned row, unsigned c16) { return row * 128u + ((c16 ^ ((row & 2u) | ((row >> 1) & 4u))) << 4); }
__device__ __forceinline__ u32x4 tr_frag(const char* img, unsigned row0, unsigned ni, unsigned lane) {
  const unsigned g = lane >> 4, q = (lane & 15u) >> 2, p = lane & 3u;
  const unsigned r0 = row0 + 8u * g + q;
  const s16x4 a = __builtin_amdgcn_ds_read_tr16_b64_v4i16((LDSP s16x4*)(img + wimg(r0, 2u * ni + (p >> 1)) + 8u * (p & 1u)));
  const s16x4 b = __builtin_amdgcn_ds_read_tr16_b64_v4i16((LDSP s16x4*)(img + wimg(r0 + 4u, 2u * ni + (p >> 1)) + 8u * (p & 1u)));
  const u32x2 ua = __builtin_bit_cast(u32x2, a), ub = __builtin_bit_cast(u32x2, b);
  return (u32x4){ua.x, ua.y, ub.x, ub.y};
}

__global__ void __launch_bounds__(512, 2) gemm_rs(Args a) {
  extern __shared__ __attribute__((aligned(16))) char smem[];
  const unsigned b = blockIdx.x;
  char* p_dst = a.ws[(b >> 3) & 7u]; char* p_my = a.my; const char* p_x = a.x; const char* p_w = a.w; char* p_out = a.out;
  unsigned p_rank = (unsigned)a.rank;
  const u64 t0 = now();
  const unsigned pf = __builtin_amdgcn_raw_buffer_load_b32(mk((const void*)((u64)__builtin_amdgcn_s_getpc() & ~255ull), (b >> 3) == 6u ? CODE_PF : 0u),
                                                           threadIdx.x * 32u, 0, 0);
  asm volatile("" : "+s"(p_dst), "+s"(p_my), "+s"(p_x), "+s"(p_w), "+s"(p_out), "+s"(p_rank));
  const unsigned lane = threadIdx.x & 63u;
  const unsigned wave = __builtin_amdgcn_readfirstlane(threadIdx.x >> 6);
  const unsigned c = 3u - ((b >> 6) & 3u), qrel = ((b >> 3) + 2u) & 7u, j = (b + qrel) & 7u;
  const unsigned rt = j >> 2, tj = 4u * c + (j & 3u), w24 = c * 8u + j;
  const bool xsnd = b < 48u, wsnd = qrel == 0u && c < 3u;
  const unsigned erel = wsnd ? 2u + w24 % 6u : qrel;
  const unsigned rank = p_rank & 7u;
  const unsigned rm = rank >> 2, rn = (rank >> 1) & 1u, rk = rank & 1u, q = rank ^ qrel;
  const rsrc_t rs_my = mk(p_my, WS_NEED), rs_d = mk(p_dst, WS_NEED), rs_x = mk(p_x, X_BYTES), rs_w = mk(p_w, W_BYTES);
  const unsigned callv = __builtin_amdgcn_raw_buffer_load_b32(rs_my, CTR_OFF + b * 4u, 0, SYS);
  unsigned call;
  unsigned t_w0 = 80u + ((XSLOTS * DS_IN) >> 8);
  auto rebase = [&]() {
    wait_until(t0, 300u);
    const unsigned an = __builtin_amdgcn_raw_buffer_load_b32(rs_my, ANC_OFF + (erel < 2u ? 2u : erel) * 128u + (lane & 7u) * 4u, 0, SYS);
    int d = (int)(an - (unsigned)t0);
    d = (d < -200 || d > 400) ? 1000 : d;
    int v = d * 256;
    #pragma unroll
    for (int o = 1; o < 8; ++o) { const int e = __shfl_xor(d, o); v -= (e < d || (e == d && ((lane ^ o) & 7u) < (lane & 7u))) ? (int)(4u * DS_IN) : 0; }
    v = d == 1000 ? -(1 << 20) : v;
    #pragma unroll
    for (int o = 1; o < 8; o <<= 1) { const int e = __shfl_xor(v, o); v = e > v ? e : v; }
    v = __builtin_amdgcn_readfirstlane(v);
    if (v > -(1 << 20)) t_w0 = (unsigned)((v + (int)(XSLOTS * DS_IN)) >> 8);
  };

  if (__builtin_expect(xsnd, 1)) {
    u32x4 xs[4];
    if (wave != 0u) __builtin_amdgcn_s_sleep(6);
    #pragma unroll
    for (int ks = 0; ks < 4; ++ks)
      xs[ks] = ld128<0>(rs_x, ((lane & 15u) * 256u + ks * 32u + (lane >> 4) * 8u) * 2u, (((q ^ wave) * 256u + (q >> 2) * 128u + j * 16u) * 256u + (q & 1u) * 128u) * 2u);
    CLOBBER();
    if (wave != 0u) {
      __builtin_amdgcn_s_barrier();
      const unsigned xbase = (unsigned)(now() - t0) - LEAD;
      if (wave == 7u && lane == 0u) __builtin_amdgcn_raw_buffer_store_b32((unsigned)t0 + xbase, rs_my, ANC_OFF + qrel * 128u + j * 4u, 0, SYS);
      t_w0 = xbase + ((XSLOTS * DS_IN) >> 8);
      wait_until(t0, xbase + (((wave - (wave > qrel ? 1u : 0u)) * 32u * DS_IN) >> 8));
    }
    CLOBBER();
    call = __builtin_amdgcn_readfirstlane(callv) + 1u;
    if (wave != qrel) {
      const unsigned dbase = (call % 3u) * ROT_BYTES + (qrel - 2u) * SLOT_BYTES + (wave * 8u + j) * 4096u + lane * 16u;
      #pragma unroll
      for (int ks = 0; ks < 4; ++ks) st128<SYS>(nz(xs[ks]), rs_d, dbase + ks * 1024u);
    }
    if (wave == 0u) {
      CLOBBER();
      __builtin_amdgcn_s_barrier();
      t_w0 = (unsigned)(now() - t0) + ((XSLOTS * DS_IN) >> 8);
    }
    wait_until(t0, t_w0);
  } else if (wsnd) {
    const unsigned d = rank ^ erel, k4 = w24 / 6u;
    const rsrc_t rs_e = mk(a.ws[(erel - 2u) & 7u], WS_NEED);
    wait_until(t0, T_LOCAL);
    u32x4 wl[2][4];
    #pragma unroll
    for (int h = 0; h < 2; ++h)
      #pragma unroll
      for (int i = 0; i < 4; ++i)
        wl[h][i] = ld128<0>(rs_w, ((lane >> 3) * 2048u + (lane & 7u) * 8u + h * 64u) * 2u,
                            (((d & 1u) * 128u + k4 * 32u + i * 8u) * 2048u + ((d >> 1) & 1u) * 1024u + wave * 128u) * 2u);
    call = __builtin_amdgcn_readfirstlane(callv) + 1u;
    const unsigned dbase = (call % 3u) * ROT_BYTES + (erel - 2u) * SLOT_BYTES + 256u * KIB + (wave * 8u + k4) * 4096u + lane * 16u;
    char* img = smem + wave * 16384u;
    #pragma unroll
    for (int h = 0; h < 2; ++h)
      #pragma unroll
      for (int i = 0; i < 4; ++i) *(LDSP u32x4*)(img + h * 4096u + wimg(i * 8u + (lane >> 3), lane & 7u)) = wl[h][i];
    #pragma unroll
    for (int h = 0; h < 2; ++h)
      #pragma unroll
      for (int ni = 0; ni < 4; ++ni) wl[h][ni] = nz(tr_frag(img + h * 4096u, 0, ni, lane));
    rebase();
    #pragma unroll
    for (int h = 0; h < 2; ++h) {
      wait_until(t0, t_w0 + (((wave * 8u + k4 + h * 4u) * 4u * DS_IN) >> 8));
      #pragma unroll
      for (int ni = 0; ni < 4; ++ni) st128<SYS>(wl[h][ni], rs_e, dbase + h * 16384u + ni * 1024u);
    }
    wait_until(t0, t_w0 + ((WSLOTS * DS_IN) >> 8));
  } else {
    call = __builtin_amdgcn_readfirstlane(callv) + 1u;
  }

  asm volatile("" :: "v"(pf));
  const unsigned rot = call % 3u;
  const bool fin = qrel == 0u || (qrel >= 2u && c < 2u);
  const unsigned omh = rm ^ (qrel >> 2), onh = rn ^ ((qrel >> 1) & 1u);
  const unsigned otj = qrel == 0u ? tj : 4u * ((c == 0u ? 2u : 0u) + (qrel & 1u)) + (j & 3u);
  const bool extra = wave + 2u == qrel;
  f32x4 acc[4][4];
  if (wave < 6u && !extra) {
    u32x4 xf[4][4], wf[4][4];
    bool dead = false;
    const unsigned sbase = rot * ROT_BYTES + wave * SLOT_BYTES;
    const unsigned xblk = (qrel * 2u + rt) * 16384u, wblk = 256u * KIB + tj * 16384u;
    const unsigned moff = (lane < 32u ? xblk : wblk) + ((lane >> 3) & 3u) * 4096u + 4064u + (lane & 7u) * 4u;
    auto poll = [&](u64 need) -> u64 {
      for (;;) {
        const u64 miss = __builtin_amdgcn_ballot_w64(__builtin_amdgcn_raw_buffer_load_b32(rs_my, moff, sbase, SYS) == 0u);
        if ((miss & need) == 0ull) return miss;
        if (now() - t0 > DEADLINE) { dead = true; return 0ull; }
        __builtin_amdgcn_s_sleep(3); CLOBBER();
      }
    };
    wait_until(t0, T_LOCAL);
    bool xok = false;
    for (;;) {
      unsigned m = ~0u;
      if (!xok) {
        const u64 miss = poll(0xffffffffull);
        #pragma unroll
        for (int i = 0; i < 16; ++i) xf[i >> 2][i & 3] = ld128<SYS>(rs_my, lane * 16u + (i & 3) * 1024u, sbase + xblk + (i >> 2) * 4096u);
        if (miss != 0ull) {
          #pragma unroll
          for (int i = 0; i < 16; ++i) m = umin2(m, min4(xf[i >> 2][i & 3]));
          if (!all_nz(m) && now() - t0 <= DEADLINE) { __builtin_amdgcn_s_sleep(4); CLOBBER(); continue; }
          poll(0xffffffff00000000ull);
        }
      }
      CLOBBER();
      #pragma unroll
      for (int i = 0; i < 16; ++i) wf[i >> 2][i & 3] = ld128<SYS>(rs_my, lane * 16u + (i & 3) * 1024u, sbase + wblk + (i >> 2) * 4096u);
      unsigned mx = ~0u;
      #pragma unroll
      for (int ks = 0; ks < 4; ++ks)
        #pragma unroll
        for (int ni = 0; ni < 4; ++ni) {
          __builtin_amdgcn_sched_barrier(0);
          #pragma unroll
          for (int mi = 0; mi < 4; ++mi) acc[mi][ni] = mfma(wf[ks][ni], xf[mi][ks], ks ? acc[mi][ni] : (f32x4){0.f, 0.f, 0.f, 0.f});
          m = umin2(m, min4(wf[ks][ni]));
          mx = umin2(mx, min4(xf[ni][ks]));
        }
      xok = all_nz(mx);
      if (xok && all_nz(m)) break;
      if (dead || now() - t0 > DEADLINE) { dead = true; break; }
    }
    if (dead) acc[0][0] = acc[1][0] = acc[2][0] = acc[3][0] = (f32x4){__builtin_nanf(""), __builtin_nanf(""), __builtin_nanf(""), __builtin_nanf("")};
  } else {
    #pragma unroll
    for (int i = 0; i < 16; ++i) acc[i >> 2][i & 3] = (f32x4){0.f, 0.f, 0.f, 0.f};
    wait_until(t0, T_LOCAL);
    const unsigned lrow = extra ? rank * 256u + omh * 128u + rt * 64u : q * 256u + rm * 128u + rt * 64u;
    const unsigned lcol = extra ? onh * 1024u + otj * 64u : rn * 1024u + tj * 64u;
    const unsigned k0 = extra ? 0u : (wave - 6u) * 128u, k1 = extra ? (fin ? 256u : 0u) : k0 + 128u;
    char* img = smem + wave * 16384u;
    const bool zer = !extra && qrel != 0u && (c == 1u || c == 2u);
    #pragma unroll 1
    for (unsigned ph = 0; ph < 2u; ++ph) {
      if ((ph ^ wave) & 1u) {
        if (zer) {
          const unsigned zi = (((qrel - 1u) * 2u + (c - 1u)) * 8u + j) * 2u + (wave - 6u);
          const unsigned zb = ((call + 2u) % 3u) * ROT_BYTES + lane * 16u;
          unsigned tz = (unsigned)(now() - t0) + (zi & 15u);
          #pragma unroll 1
          for (unsigned i = zi; i < ROT_BYTES / 1024u; i += 224u) {
            wait_until(t0, tz); tz += Z_STEP;
            st128<DEV>((u32x4){0u, 0u, 0u, 0u}, rs_my, zb + i * 1024u);
          }
        }
      } else {
        #pragma unroll 1
        for (unsigned k = k0; k < k1; k += 128u) {
          u32x4 wr[16], xf[4][4];
          #pragma unroll
          for (int i = 0; i < 16; ++i) wr[i] = ld128<0>(rs_w, ((lane >> 3) * 2048u + (lane & 7u) * 8u) * 2u, ((k + i * 8u) * 2048u + lcol) * 2u);
          #pragma unroll
          for (int mi = 0; mi < 4; ++mi)
            #pragma unroll
            for (int ks = 0; ks < 4; ++ks)
              xf[mi][ks] = ld128<0>(rs_x, ((lane & 15u) * 256u + ks * 32u + (lane >> 4) * 8u) * 2u, ((lrow + mi * 16u) * 256u + k) * 2u);
          __builtin_amdgcn_sched_barrier(0);
          #pragma unroll
          for (int i = 0; i < 16; ++i) *(LDSP u32x4*)(img + wimg(i * 8u + (lane >> 3), lane & 7u)) = wr[i];
          #pragma unroll
          for (int ks = 0; ks < 4; ++ks)
            #pragma unroll
            for (int ni = 0; ni < 4; ++ni) {
              const u32x4 wf = tr_frag(img, ks * 32u, ni, lane);
              #pragma unroll
              for (int mi = 0; mi < 4; ++mi) acc[mi][ni] = mfma(wf, xf[mi][ks], acc[mi][ni]);
            }
          CLOBBER();
        }
      }
    }
    if (wave == 7u && qrel != 0u) { rebase(); if (lane == 0u) *(LDSP unsigned*)(smem + LDS_TW) = t_w0 + (unsigned)t0; }
  }
  {
    char* slab = smem + wave * 16384u;
    #pragma unroll
    for (int mi = 0; mi < 4; ++mi)
      #pragma unroll
      for (int ni = 0; ni < 4; ++ni) {
        const unsigned nq = ni * 4u + (lane >> 4);
        *(LDSP f32x4*)(slab + (((mi * 16u + nq) * 16u + ((lane & 15u) ^ (nq & 14u))) << 4)) = acc[mi][ni];
      }
  }
  asm volatile("s_waitcnt lgkmcnt(0)" ::: "memory");
  __builtin_amdgcn_s_barrier();
  CLOBBER();
  if (threadIdx.x == 0) __builtin_amdgcn_raw_buffer_store_b32(call, rs_my, CTR_OFF + b * 4u, 0, 0);
  f32x4 s0, s1, e0, e1;
  {
    const unsigned m = wave * 8u + (lane >> 3), o = lane & 7u;
    const unsigned ad = (((m >> 4) * 16u + 2u * o) * 16u + ((m & 15u) ^ (2u * o))) << 4;
    const unsigned sx = (qrel - 2u) & 7u;
    e0 = *(LDSP f32x4*)(smem + sx * 16384u + ad);
    e1 = *(LDSP f32x4*)(smem + sx * 16384u + ad + 256u);
    s0 = *(LDSP f32x4*)(smem + (sx ^ 1u) * 16384u + ad);
    s1 = *(LDSP f32x4*)(smem + (sx ^ 1u) * 16384u + ad + 256u);
    #pragma unroll
    for (int s = 2; s < 8; ++s) {
      s0 += *(LDSP f32x4*)(smem + (sx ^ s) * 16384u + ad);
      s1 += *(LDSP f32x4*)(smem + (sx ^ s) * 16384u + ad + 256u);
    }
    if (qrel < 2u) { s0 += e0; s1 += e1; e0 = s0; e1 = s1; }
  }
  const unsigned strip = wave * 1024u + lane * 16u;
  if (qrel != 0u) {
    const u32x4 p = {pack2(s0.x, s0.y), pack2(s0.z, s0.w), pack2(s1.x, s1.y), pack2(s1.z, s1.w)};
    const unsigned tb = __builtin_amdgcn_readfirstlane(*(LDSP unsigned*)(smem + LDS_TW)) - (unsigned)t0 + ((WSLOTS * DS_IN) >> 8) - (qrel == 1u ? 60u : 0u);
    wait_until(t0, tb + ((((tj * 2u + rt) * 8u + wave) * DS_OUT) >> 8));
    st128<SYS>(nz(p), rs_d, strip + rot * ROT_BYTES + IN_BYTES + rk * MIB + (((rm * 2u + rt) * 32u + rn * 16u + tj) << 13));
  }
  if (fin) {
    const unsigned tbase = rot * ROT_BYTES + IN_BYTES + (((omh * 2u + rt) * 32u + onh * 16u + otj) << 13);
    const unsigned pa = tbase + (rk ^ 1u) * MIB, pb = tbase + (qrel != 0u ? rk : (rk ^ 1u)) * MIB;
    const rsrc_t rs_o = mk(p_out, Y_BYTES);
    unsigned ooff = ((omh * 128u + rt * 64u + wave * 8u + (lane >> 3)) * 2048u + onh * 1024u + otj * 64u + (lane & 7u) * 8u) * 2u;
    asm volatile("" : "+v"(ooff));
    const unsigned vmask = qrel != 0u ? ~0u : 0u;
    const bool wt = otj < 12u && qrel != 0u;
    for (unsigned it = 0;; ++it) {
      const u32x4 u = ld128<SYS>(rs_my, strip, pa);
      u32x4 v = ld128<SYS>(rs_my, strip, pb);
      const bool ok = all_nz(umin2(min4(u), min4(v)));
      if (ok || it == 0u) {
        v &= vmask;
        const float y0 = e0.x + lo16(u.x) + lo16(v.x), y1 = e0.y + hi16(u.x) + hi16(v.x), y2 = e0.z + lo16(u.y) + lo16(v.y), y3 = e0.w + hi16(u.y) + hi16(v.y);
        const float y4 = e1.x + lo16(u.z) + lo16(v.z), y5 = e1.y + hi16(u.z) + hi16(v.z), y6 = e1.z + lo16(u.w) + lo16(v.w), y7 = e1.w + hi16(u.w) + hi16(v.w);
        const u32x4 o = {pack2(silu(y0), silu(y1)), pack2(silu(y2), silu(y3)), pack2(silu(y4), silu(y5)), pack2(silu(y6), silu(y7))};
        const unsigned off = ok ? ooff : Y_BYTES;
        if (wt) st128<DEV>(o, rs_o, off); else st128<0>(o, rs_o, off);
        if (ok) break;
      }
      if ((it & 63u) == 63u && now() - t0 > DEADLINE) { st128<0>((u32x4){0x7fc07fc0u, 0x7fc07fc0u, 0x7fc07fc0u, 0x7fc07fc0u}, rs_o, ooff); break; }
      __builtin_amdgcn_s_sleep(1);
      CLOBBER();
    }
  }
}

extern "C" void kernel_launch(int rank, int nranks, ncclComm_t comm, hipStream_t stream,
                              void* const* d_in, const int* in_sizes, int n_in,
                              void* d_out, int out_size, const int* in_dtypes, int out_dtype,
                              void* const* d_out_all, void* const* d_ws_all, size_t ws_bytes) {
  (void)comm; (void)in_dtypes; (void)out_dtype; (void)d_out_all;
  if (nranks != 8 || (unsigned)rank >= 8u || n_in < 2 || ws_bytes < WS_NEED || (size_t)in_sizes[0] * 2 != X_BYTES ||
      (size_t)in_sizes[1] * 2 != W_BYTES || (size_t)out_size * 2 != Y_BYTES) {
    fprintf(stderr, "kernel_launch: unexpected configuration\n");
    return;
  }
  const int lds = LDS_BYTES;
  static bool attr_set = false;
  if (!attr_set) {
    hipError_t e = hipFuncSetAttribute((const void*)gemm_rs, hipFuncAttributeMaxDynamicSharedMemorySize, lds);
    if (e != hipSuccess) { fprintf(stderr, "hipFuncSetAttribute: %s\n", hipGetErrorString(e)); return; }
    attr_set = true;
  }
  Args a;
  for (int p = 0; p < 8; ++p) a.ws[p] = (char*)d_ws_all[rank ^ ((p + 2) & 7)];
  a.my = (char*)d_ws_all[rank]; a.x = (const char*)d_in[0]; a.w = (const char*)d_in[1]; a.out = (char*)d_out; a.rank = rank;
  gemm_rs<<<256, 512, lds, stream>>>(a);
}
```

```cpp
#include <hip/hip_runtime.h>
#include <rccl/rccl.h>
#include <cstdio>

typedef unsigned u32x4 __attribute__((ext_vector_type(4)));
typedef unsigned u32x2 __attribute__((ext_vector_type(2)));
typedef short s16x4 __attribute__((ext_vector_type(4)));
typedef __bf16 bf16x8 __attribute__((ext_vector_type(8)));
typedef __bf16 bf16x2 __attribute__((ext_vector_type(2)));
typedef float f32x2 __attribute__((ext_vector_type(2)));
typedef float f32x4 __attribute__((ext_vector_type(4)));
typedef unsigned long long u64;
typedef __amdgpu_buffer_rsrc_t rsrc_t;
#define LDSP __attribute__((address_space(3)))
#define CLOBBER() asm volatile("" ::: "memory")

constexpr unsigned KIB = 1024u, MIB = 1024u * 1024u;
constexpr unsigned SLOT_BYTES = 512 * KIB;
constexpr unsigned IN_BYTES = 6 * SLOT_BYTES;
constexpr unsigned PART_BYTES = 2 * MIB;
constexpr unsigned ROT_BYTES = IN_BYTES + PART_BYTES;
constexpr unsigned CTR_OFF = 3 * ROT_BYTES;
constexpr unsigned ANC_OFF = CTR_OFF + 4096u;
constexpr unsigned TMO_OFF = CTR_OFF + 8192u;
constexpr unsigned WS_NEED = CTR_OFF + 64 * KIB;
constexpr unsigned X_BYTES = 2048u * 256u * 2u, W_BYTES = 256u * 2048u * 2u, Y_BYTES = 256u * 2048u * 2u;
constexpr u64 DEADLINE = 100000000ull;
constexpr unsigned DS_IN = 434u, DS_OUT = 438u;
constexpr unsigned XSLOTS = 224u, WSLOTS = 256u;
constexpr unsigned LEAD = 20u;
constexpr unsigned T_LOCAL = 120u;
constexpr unsigned Z_STEP = 12u;
constexpr unsigned CODE_PF = 11904u;
constexpr int SYS = 17, DEV = 16;
constexpr int LDS_TW = 128 * 1024, LDS_BYTES = LDS_TW + 64;

struct Args {
  const char* x; char* my; const char* w; char* out;
  long long rank;
  char* ws[8];
};

__device__ __forceinline__ u64 now() { return __builtin_amdgcn_s_memrealtime(); }
__device__ __forceinline__ rsrc_t mk(const void* p, unsigned bytes) { return __builtin_amdgcn_make_buffer_rsrc((void*)p, 0, bytes, 0x00020000); }
__device__ __forceinline__ unsigned umin2(unsigned a, unsigned b) { return a < b ? a : b; }
__device__ __forceinline__ unsigned min4(u32x4 v) { return umin2(umin2(v.x, v.y), umin2(v.z, v.w)); }
__device__ __forceinline__ bool all_nz(unsigned m) { return __builtin_amdgcn_ballot_w64(m == 0u) == 0ull; }
__device__ __forceinline__ u32x4 nz(u32x4 v) {
  v.x = v.x ? v.x : 0x8000u; v.y = v.y ? v.y : 0x8000u; v.z = v.z ? v.z : 0x8000u; v.w = v.w ? v.w : 0x8000u; return v;
}
__device__ __forceinline__ unsigned pack2(float a, float b) { f32x2 f = {a, b}; return __builtin_bit_cast(unsigned, __builtin_convertvector(f, bf16x2)); }
__device__ __forceinline__ float lo16(unsigned u) { return __builtin_bit_cast(float, u << 16); }
__device__ __forceinline__ float hi16(unsigned u) { return __builtin_bit_cast(float, u & 0xffff0000u); }
__device__ __forceinline__ float silu(float y) {
  const float c = __builtin_amdgcn_fmed3f(y, -60.f, 60.f);
  const float e = __builtin_amdgcn_exp2f(-1.4426950408889634f * c);
  return y * __builtin_amdgcn_rcpf(1.f + e);
}
template <int AUX> __device__ __forceinline__ void st128(u32x4 v, rsrc_t r, unsigned off) { __builtin_amdgcn_raw_buffer_store_b128(v, r, off, 0, AUX); }
template <int AUX> __device__ __forceinline__ u32x4 ld128(rsrc_t r, unsigned voff, unsigned soff) { return __builtin_amdgcn_raw_buffer_load_b128(r, voff, soff, AUX); }
__device__ __forceinline__ void wait_until(u64 t0, unsigned ticks) {
  for (;;) {
    const unsigned el = (unsigned)(now() - t0);
    if (el >= ticks) break;
    if (ticks - el > 60u) __builtin_amdgcn_s_sleep(16); else __builtin_amdgcn_s_sleep(1);
  }
}
__device__ __forceinline__ f32x4 mfma(u32x4 wf, u32x4 xf, f32x4 c) {
  return __builtin_amdgcn_mfma_f32_16x16x32_bf16(__builtin_bit_cast(bf16x8, wf), __builtin_bit_cast(bf16x8, xf), c, 0, 0, 0);
}
__device__ __forceinline__ unsigned wimg(unsigned row, unsigned c16) { return row * 128u + ((c16 ^ ((row & 2u) | ((row >> 1) & 4u))) << 4); }
__device__ __forceinline__ u32x4 tr_frag(const char* img, unsigned row0, unsigned ni, unsigned lane) {
  const unsigned g = lane >> 4, q = (lane & 15u) >> 2, p = lane & 3u;
  const unsigned r0 = row0 + 8u * g + q;
  const s16x4 a = __builtin_amdgcn_ds_read_tr16_b64_v4i16((LDSP s16x4*)(img + wimg(r0, 2u * ni + (p >> 1)) + 8u * (p & 1u)));
  const s16x4 b = __builtin_amdgcn_ds_read_tr16_b64_v4i16((LDSP s16x4*)(img + wimg(r0 + 4u, 2u * ni + (p >> 1)) + 8u * (p & 1u)));
  const u32x2 ua = __builtin_bit_cast(u32x2, a), ub = __builtin_bit_cast(u32x2, b);
  return (u32x4){ua.x, ua.y, ub.x, ub.y};
}

__global__ void __launch_bounds__(512, 2) gemm_rs(Args a) {
  extern __shared__ __attribute__((aligned(16))) char smem[];
  const unsigned b = blockIdx.x;
  char* p_dst = a.ws[(b >> 3) & 7u]; char* p_my = a.my; const char* p_x = a.x; const char* p_w = a.w; char* p_out = a.out;
  unsigned p_rank = (unsigned)a.rank;
  const u64 t0 = now();
  const unsigned pf = __builtin_amdgcn_raw_buffer_load_b32(mk((const void*)((u64)__builtin_amdgcn_s_getpc() & ~255ull), (b >> 3) == 6u ? CODE_PF : 0u),
                                                           threadIdx.x * 32u, 0, 0);
  asm volatile("" : "+s"(p_dst), "+s"(p_my), "+s"(p_x), "+s"(p_w), "+s"(p_out), "+s"(p_rank));
  const unsigned lane = threadIdx.x & 63u;
  const unsigned wave = __builtin_amdgcn_readfirstlane(threadIdx.x >> 6);
  const unsigned c = 3u - ((b >> 6) & 3u), qrel = ((b >> 3) + 2u) & 7u, j = (b + qrel) & 7u;
  const unsigned rt = j >> 2, tj = 4u * c + (j & 3u), w24 = c * 8u + j;
  const bool xsnd = b < 48u, wsnd = qrel == 0u && c < 3u;
  const unsigned erel = wsnd ? 2u + w24 % 6u : qrel;
  const unsigned rank = p_rank & 7u;
  const unsigned rm = rank >> 2, rn = (rank >> 1) & 1u, rk = rank & 1u, q = rank ^ qrel;
  const rsrc_t rs_my = mk(p_my, WS_NEED), rs_d = mk(p_dst, WS_NEED), rs_x = mk(p_x, X_BYTES), rs_w = mk(p_w, W_BYTES);
  const unsigned callv = __builtin_amdgcn_raw_buffer_load_b32(rs_my, CTR_OFF + b * 4u, 0, SYS);
  unsigned call;
  unsigned t_w0 = 80u + ((XSLOTS * DS_IN) >> 8);
  auto rebase = [&]() {
    wait_until(t0, 300u);
    const unsigned an = __builtin_amdgcn_raw_buffer_load_b32(rs_my, ANC_OFF + (erel < 2u ? 2u : erel) * 128u + (lane & 7u) * 4u, 0, SYS);
    int d = (int)(an - (unsigned)t0);
    d = (d < -200 || d > 400) ? 1000 : d;
    int v = d * 256;
    #pragma unroll
    for (int o = 1; o < 8; ++o) { const int e = __shfl_xor(d, o); v -= (e < d || (e == d && ((lane ^ o) & 7u) < (lane & 7u))) ? (int)(4u * DS_IN) : 0; }
    v = d == 1000 ? -(1 << 20) : v;
    #pragma unroll
    for (int o = 1; o < 8; o <<= 1) { const int e = __shfl_xor(v, o); v = e > v ? e : v; }
    v = __builtin_amdgcn_readfirstlane(v);
    if (v > -(1 << 20)) t_w0 = (unsigned)((v + (int)(XSLOTS * DS_IN)) >> 8);
  };

  if (__builtin_expect(xsnd, 1)) {
    u32x4 xs[4];
    if (wave != 0u) __builtin_amdgcn_s_sleep(6);
    #pragma unroll
    for (int ks = 0; ks < 4; ++ks)
      xs[ks] = ld128<0>(rs_x, ((lane & 15u) * 256u + ks * 32u + (lane >> 4) * 8u) * 2u, (((q ^ wave) * 256u + (q >> 2) * 128u + j * 16u) * 256u + (q & 1u) * 128u) * 2u);
    CLOBBER();
    if (wave != 0u) {
      __builtin_amdgcn_s_barrier();
      const unsigned xbase = (unsigned)(now() - t0) - LEAD;
      if (wave == 7u && lane == 0u) __builtin_amdgcn_raw_buffer_store_b32((unsigned)t0 + xbase, rs_my, ANC_OFF + qrel * 128u + j * 4u, 0, SYS);
      t_w0 = xbase + ((XSLOTS * DS_IN) >> 8);
      wait_until(t0, xbase + (((wave - (wave > qrel ? 1u : 0u)) * 32u * DS_IN) >> 8));
    }
    CLOBBER();
    call = __builtin_amdgcn_readfirstlane(callv) + 1u;
    if (wave != qrel) {
      const unsigned dbase = (call % 3u) * ROT_BYTES + (qrel - 2u) * SLOT_BYTES + (wave * 8u + j) * 4096u + lane * 16u;
      #pragma unroll
      for (int ks = 0; ks < 4; ++ks) st128<SYS>(nz(xs[ks]), rs_d, dbase + ks * 1024u);
    }
    if (wave == 0u) {
      CLOBBER();
      __builtin_amdgcn_s_barrier();
      t_w0 = (unsigned)(now() - t0) + ((XSLOTS * DS_IN) >> 8);
    }
    wait_until(t0, t_w0);
  } else if (wsnd) {
    const unsigned d = rank ^ erel, k4 = w24 / 6u;
    const rsrc_t rs_e = mk(a.ws[(erel - 2u) & 7u], WS_NEED);
    wait_until(t0, T_LOCAL);
    u32x4 wl[2][4];
    #pragma unroll
    for (int h = 0; h < 2; ++h)
      #pragma unroll
      for (int i = 0; i < 4; ++i)
        wl[h][i] = ld128<0>(rs_w, ((lane >> 3) * 2048u + (lane & 7u) * 8u + h * 64u) * 2u,
                            (((d & 1u) * 128u + k4 * 32u + i * 8u) * 2048u + ((d >> 1) & 1u) * 1024u + wave * 128u) * 2u);
    call = __builtin_amdgcn_readfirstlane(callv) + 1u;
    const unsigned dbase = (call % 3u) * ROT_BYTES + (erel - 2u) * SLOT_BYTES + 256u * KIB + (wave * 8u + k4) * 4096u + lane * 16u;
    char* img = smem + wave * 16384u;
    #pragma unroll
    for (int h = 0; h < 2; ++h)
      #pragma unroll
      for (int i = 0; i < 4; ++i) *(LDSP u32x4*)(img + h * 4096u + wimg(i * 8u + (lane >> 3), lane & 7u)) = wl[h][i];
    #pragma unroll
    for (int h = 0; h < 2; ++h)
      #pragma unroll
      for (int ni = 0; ni < 4; ++ni) wl[h][ni] = nz(tr_frag(img + h * 4096u, 0, ni, lane));
    rebase();
    #pragma unroll
    for (int h = 0; h < 2; ++h) {
      wait_until(t0, t_w0 + (((wave * 8u + k4 + h * 4u) * 4u * DS_IN) >> 8));
      #pragma unroll
      for (int ni = 0; ni < 4; ++ni) st128<SYS>(wl[h][ni], rs_e, dbase + h * 16384u + ni * 1024u);
    }
    wait_until(t0, t_w0 + ((WSLOTS * DS_IN) >> 8));
  } else {
    call = __builtin_amdgcn_readfirstlane(callv) + 1u;
  }

  asm volatile("" :: "v"(pf));
  const unsigned rot = call % 3u;
  const bool fin = qrel == 0u || (qrel >= 2u && c < 2u);
  const unsigned omh = rm ^ (qrel >> 2), onh = rn ^ ((qrel >> 1) & 1u);
  const unsigned otj = qrel == 0u ? tj : 4u * ((c == 0u ? 2u : 0u) + (qrel & 1u)) + (j & 3u);
  const bool extra = wave + 2u == qrel;
  f32x4 acc[4][4];
  if (wave < 6u && !extra) {
    u32x4 xf[4][4], wf[4][4];
    bool dead = false;
    const unsigned sbase = rot * ROT_BYTES + wave * SLOT_BYTES;
    const unsigned xblk = (qrel * 2u + rt) * 16384u, wblk = 256u * KIB + tj * 16384u;
    const unsigned moff = (lane < 32u ? xblk : wblk) + ((lane >> 3) & 3u) * 4096u + 4064u + (lane & 7u) * 4u;
    auto poll = [&](u64 need) -> u64 {
      for (;;) {
        const u64 miss = __builtin_amdgcn_ballot_w64(__builtin_amdgcn_raw_buffer_load_b32(rs_my, moff, sbase, SYS) == 0u);
        if ((miss & need) == 0ull) return miss;
        if (now() - t0 > DEADLINE) { dead = true; return 0ull; }
        __builtin_amdgcn_s_sleep(3); CLOBBER();
      }
    };
    wait_until(t0, T_LOCAL);
    bool xok = false;
    for (;;) {
      unsigned m = ~0u;
      if (!xok) {
        const u64 miss = poll(0xffffffffull);
        #pragma unroll
        for (int i = 0; i < 16; ++i) xf[i >> 2][i & 3] = ld128<SYS>(rs_my, lane * 16u + (i & 3) * 1024u, sbase + xblk + (i >> 2) * 4096u);
        if (miss != 0ull) {
          #pragma unroll
          for (int i = 0; i < 16; ++i) m = umin2(m, min4(xf[i >> 2][i & 3]));
          if (!all_nz(m) && now() - t0 <= DEADLINE) { __builtin_amdgcn_s_sleep(4); CLOBBER(); continue; }
          poll(0xffffffff00000000ull);
        }
      }
      CLOBBER();
      #pragma unroll
      for (int i = 0; i < 16; ++i) wf[i >> 2][i & 3] = ld128<SYS>(rs_my, lane * 16u + (i & 3) * 1024u, sbase + wblk + (i >> 2) * 4096u);
      unsigned mx = ~0u;
      #pragma unroll
      for (int ks = 0; ks < 4; ++ks)
        #pragma unroll
        for (int ni = 0; ni < 4; ++ni) {
          __builtin_amdgcn_sched_barrier(0);
          #pragma unroll
          for (int mi = 0; mi < 4; ++mi) acc[mi][ni] = mfma(wf[ks][ni], xf[mi][ks], ks ? acc[mi][ni] : (f32x4){0.f, 0.f, 0.f, 0.f});
          m = umin2(m, min4(wf[ks][ni]));
          mx = umin2(mx, min4(xf[ni][ks]));
        }
      xok = all_nz(mx);
      if (xok && all_nz(m)) break;
      if (dead || now() - t0 > DEADLINE) { dead = true; break; }
    }
    if (dead) {
      acc[0][0] = acc[1][0] = acc[2][0] = acc[3][0] = (f32x4){__builtin_nanf(""), __builtin_nanf(""), __builtin_nanf(""), __builtin_nanf("")};
      __builtin_amdgcn_raw_buffer_store_b32(call, rs_my, TMO_OFF, 0, DEV);
    }
  } else {
    #pragma unroll
    for (int i = 0; i < 16; ++i) acc[i >> 2][i & 3] = (f32x4){0.f, 0.f, 0.f, 0.f};
    wait_until(t0, T_LOCAL);
    const unsigned lrow = extra ? rank * 256u + omh * 128u + rt * 64u : q * 256u + rm * 128u + rt * 64u;
    const unsigned lcol = extra ? onh * 1024u + otj * 64u : rn * 1024u + tj * 64u;
    const unsigned k0 = extra ? 0u : (wave - 6u) * 128u, k1 = extra ? (fin ? 256u : 0u) : k0 + 128u;
    char* img = smem + wave * 16384u;
    const bool zer = !extra && qrel != 0u && (c == 1u || c == 2u);
    #pragma unroll 1
    for (unsigned ph = 0; ph < 2u; ++ph) {
      if ((ph ^ wave) & 1u) {
        if (zer) {
          const unsigned zi = (((qrel - 1u) * 2u + (c - 1u)) * 8u + j) * 2u + (wave - 6u);
          const unsigned zb = ((call + 2u) % 3u) * ROT_BYTES + lane * 16u;
          unsigned tz = (unsigned)(now() - t0) + (zi & 15u);
          #pragma unroll 1
          for (unsigned i = zi; i < ROT_BYTES / 1024u; i += 224u) {
            wait_until(t0, tz); tz += Z_STEP;
            st128<DEV>((u32x4){0u, 0u, 0u, 0u}, rs_my, zb + i * 1024u);
          }
        }
      } else {
        #pragma unroll 1
        for (unsigned k = k0; k < k1; k += 128u) {
          u32x4 wr[16], xf[4][4];
          #pragma unroll
          for (int i = 0; i < 16; ++i) wr[i] = ld128<0>(rs_w, ((lane >> 3) * 2048u + (lane & 7u) * 8u) * 2u, ((k + i * 8u) * 2048u + lcol) * 2u);
          #pragma unroll
          for (int mi = 0; mi < 4; ++mi)
            #pragma unroll
            for (int ks = 0; ks < 4; ++ks)
              xf[mi][ks] = ld128<0>(rs_x, ((lane & 15u) * 256u + ks * 32u + (lane >> 4) * 8u) * 2u, ((lrow + mi * 16u) * 256u + k) * 2u);
          __builtin_amdgcn_sched_barrier(0);
          #pragma unroll
          for (int i = 0; i < 16; ++i) *(LDSP u32x4*)(img + wimg(i * 8u + (lane >> 3), lane & 7u)) = wr[i];
          #pragma unroll
          for (int ks = 0; ks < 4; ++ks)
            #pragma unroll
            for (int ni = 0; ni < 4; ++ni) {
              const u32x4 wf = tr_frag(img, ks * 32u, ni, lane);
              #pragma unroll
              for (int mi = 0; mi < 4; ++mi) acc[mi][ni] = mfma(wf, xf[mi][ks], acc[mi][ni]);
            }
          CLOBBER();
        }
      }
    }
    if (wave == 7u && qrel != 0u) { rebase(); if (lane == 0u) *(LDSP unsigned*)(smem + LDS_TW) = t_w0 + (unsigned)t0; }
  }
  {
    char* slab = smem + wave * 16384u;
    #pragma unroll
    for (int mi = 0; mi < 4; ++mi)
      #pragma unroll
      for (int ni = 0; ni < 4; ++ni) {
        const unsigned nq = ni * 4u + (lane >> 4);
        *(LDSP f32x4*)(slab + (((mi * 16u + nq) * 16u + ((lane & 15u) ^ (nq & 14u))) << 4)) = acc[mi][ni];
      }
  }
  asm volatile("s_waitcnt lgkmcnt(0)" ::: "memory");
  __builtin_amdgcn_s_barrier();
  CLOBBER();
  if (threadIdx.x == 0) __builtin_amdgcn_raw_buffer_store_b32(call, rs_my, CTR_OFF + b * 4u, 0, 0);
  f32x4 s0, s1, e0, e1;
  {
    const unsigned m = wave * 8u + (lane >> 3), o = lane & 7u;
    const unsigned ad = (((m >> 4) * 16u + 2u * o) * 16u + ((m & 15u) ^ (2u * o))) << 4;
    const unsigned sx = (qrel - 2u) & 7u;
    e0 = *(LDSP f32x4*)(smem + sx * 16384u + ad);
    e1 = *(LDSP f32x4*)(smem + sx * 16384u + ad + 256u);
    s0 = *(LDSP f32x4*)(smem + (sx ^ 1u) * 16384u + ad);
    s1 = *(LDSP f32x4*)(smem + (sx ^ 1u) * 16384u + ad + 256u);
    #pragma unroll
    for (int s = 2; s < 8; ++s) {
      s0 += *(LDSP f32x4*)(smem + (sx ^ s) * 16384u + ad);
      s1 += *(LDSP f32x4*)(smem + (sx ^ s) * 16384u + ad + 256u);
    }
    if (qrel < 2u) { s0 += e0; s1 += e1; e0 = s0; e1 = s1; }
  }
  const unsigned strip = wave * 1024u + lane * 16u;
  if (qrel != 0u) {
    const u32x4 p = {pack2(s0.x, s0.y), pack2(s0.z, s0.w), pack2(s1.x, s1.y), pack2(s1.z, s1.w)};
    const unsigned tb = __builtin_amdgcn_readfirstlane(*(LDSP unsigned*)(smem + LDS_TW)) - (unsigned)t0 + ((WSLOTS * DS_IN) >> 8) - (qrel == 1u ? 60u : 0u);
    wait_until(t0, tb + ((((tj * 2u + rt) * 8u + wave) * DS_OUT) >> 8));
    st128<SYS>(nz(p), rs_d, strip + rot * ROT_BYTES + IN_BYTES + rk * MIB + (((rm * 2u + rt) * 32u + rn * 16u + tj) << 13));
  }
  if (fin) {
    const unsigned tbase = rot * ROT_BYTES + IN_BYTES + (((omh * 2u + rt) * 32u + onh * 16u + otj) << 13);
    const unsigned pa = tbase + (rk ^ 1u) * MIB, pb = tbase + (qrel != 0u ? rk : (rk ^ 1u)) * MIB;
    const rsrc_t rs_o = mk(p_out, Y_BYTES);
    unsigned ooff = ((omh * 128u + rt * 64u + wave * 8u + (lane >> 3)) * 2048u + onh * 1024u + otj * 64u + (lane & 7u) * 8u) * 2u;
    asm volatile("" : "+v"(ooff));
    const unsigned vmask = qrel != 0u ? ~0u : 0u;
    const bool wt = otj < 12u && qrel != 0u;
    for (unsigned it = 0;; ++it) {
      const u32x4 u = ld128<SYS>(rs_my, strip, pa);
      u32x4 v = ld128<SYS>(rs_my, strip, pb);
      const bool ok = all_nz(umin2(min4(u), min4(v)));
      if (ok || it == 0u) {
        v &= vmask;
        const float y0 = e0.x + lo16(u.x) + lo16(v.x), y1 = e0.y + hi16(u.x) + hi16(v.x), y2 = e0.z + lo16(u.y) + lo16(v.y), y3 = e0.w + hi16(u.y) + hi16(v.y);
        const float y4 = e1.x + lo16(u.z) + lo16(v.z), y5 = e1.y + hi16(u.z) + hi16(v.z), y6 = e1.z + lo16(u.w) + lo16(v.w), y7 = e1.w + hi16(u.w) + hi16(v.w);
        const u32x4 o = {pack2(silu(y0), silu(y1)), pack2(silu(y2), silu(y3)), pack2(silu(y4), silu(y5)), pack2(silu(y6), silu(y7))};
        const unsigned off = ok ? ooff : Y_BYTES;
        if (wt) st128<DEV>(o, rs_o, off); else st128<0>(o, rs_o, off);
        if (ok) break;
      }
      if ((it & 63u) == 63u && now() - t0 > DEADLINE) {
        st128<0>((u32x4){0x7fc07fc0u, 0x7fc07fc0u, 0x7fc07fc0u, 0x7fc07fc0u}, rs_o, ooff);
        __builtin_amdgcn_raw_buffer_store_b32(call, rs_my, TMO_OFF, 0, DEV);
        break;
      }
      __builtin_amdgcn_s_sleep(1);
      CLOBBER();
    }
  }
}

extern "C" void kernel_launch(int rank, int nranks, ncclComm_t comm, hipStream_t stream,
                              void* const* d_in, const int* in_sizes, int n_in,
                              void* d_out, int out_size, const int* in_dtypes, int out_dtype,
                              void* const* d_out_all, void* const* d_ws_all, size_t ws_bytes) {
  (void)comm; (void)in_dtypes; (void)out_dtype; (void)d_out_all;
  if (nranks != 8 || (unsigned)rank >= 8u || n_in < 2 || ws_bytes < WS_NEED || (size_t)in_sizes[0] * 2 != X_BYTES ||
      (size_t)in_sizes[1] * 2 != W_BYTES || (size_t)out_size * 2 != Y_BYTES) {
    fprintf(stderr, "kernel_launch: unexpected configuration\n");
    return;
  }
  const int lds = LDS_BYTES;
  static bool attr_set = false;
  if (!attr_set) {
    hipError_t e = hipFuncSetAttribute((const void*)gemm_rs, hipFuncAttributeMaxDynamicSharedMemorySize, lds);
    if (e != hipSuccess) { fprintf(stderr, "hipFuncSetAttribute: %s\n", hipGetErrorString(e)); return; }
    attr_set = true;
  }
  Args a;
  for (int p = 0; p < 8; ++p) a.ws[p] = (char*)d_ws_all[rank ^ ((p + 2) & 7)];
  a.my = (char*)d_ws_all[rank]; a.x = (const char*)d_in[0]; a.w = (const char*)d_in[1]; a.out = (char*)d_out; a.rank = rank;
  gemm_rs<<<256, 512, lds, stream>>>(a);
}
```
